```python
import numpy as np

kernel_name = "gpu_ag_gemm_m4096_k4096_n16384_bf16_qe4m3"


ABSMAX_RTOL = 0.09

COMPUTE_DTYPE = "bfloat16"
EPILOGUE = "quant_e4m3"
IN_QUANT = None
OUT_QUANT = "e4m3"
IN_DTYPES = {"x": "bfloat16", "w_mat": "bfloat16"}
WORLD_SIZE = 8

def _to_bf16(x):
    u = np.ascontiguousarray(x, dtype=np.float32).view(np.uint32)
    return ((u + 0x7FFF + ((u >> 16) & 1)) & 0xFFFF0000).view(np.float32)

def setup_inputs():

    x = _to_bf16(np.random.randn(4096, 4096).astype(np.float32))
    w_mat = _to_bf16((np.random.randn(4096, 16384) / np.sqrt(4096)).astype(np.float32))
    return {"x": x, "w_mat": w_mat}

def _snap_fp8(x, mant_bits, max_abs, min_normal):
    a = np.abs(x.astype(np.float64))
    _m, _e = np.frexp(a)
    q = np.where(a >= min_normal, np.ldexp(1.0, _e - 1 - mant_bits),
                 min_normal * 2.0 ** -mant_bits)
    snapped = np.minimum(np.round(a / q) * q, max_abs)
    return (np.sign(x) * snapped).astype(np.float32)
def _quant_dequant(y, amax):
    scale = amax / 448.0
    q = _snap_fp8(y / scale, 3, 448.0, 2.0 ** -6)
    return (q * scale).astype(np.float32)
def _epilogue(y):
    return _quant_dequant(y, np.abs(y).max())

def reference(x, w_mat):
    y = (x @ w_mat).astype(np.float32)
    return _epilogue(y)
```

```cpp
#include <hip/hip_runtime.h>
#include <rccl/rccl.h>
#include <cstdio>
#include <cstdint>

#define HIP_CHECK(expr) do { hipError_t _e = (expr); if (_e != hipSuccess) { \
    fprintf(stderr, "HIP error %s at %s:%d\n", hipGetErrorString(_e), __FILE__, __LINE__); \
    return; } } while (0)

namespace {

typedef unsigned u32;
typedef unsigned long long u64;
typedef u32 u32x2 __attribute__((ext_vector_type(2)));
typedef u32 u32x3 __attribute__((ext_vector_type(3)));
typedef u32 u32x4 __attribute__((ext_vector_type(4)));
typedef float f32x16 __attribute__((ext_vector_type(16)));
typedef __bf16 bf16x4 __attribute__((ext_vector_type(4)));
typedef __bf16 bf16x8 __attribute__((ext_vector_type(8)));
#define LDSP __attribute__((address_space(3)))
#define GLBP __attribute__((address_space(1)))

#define CFG_D 2
#define CFG_P 4
#define CFG_FORCE_RAW 0

constexpr int NR = 8;
constexpr int KD = 4096, NLOC = 2048, MLOC = 512;
constexpr int STEPS = 64;
constexpr int NWG = 256;
constexpr u32 WB_BYTES = 5888;
constexpr u32 STEP_BYTES = 8 * WB_BYTES;
constexpr u32 XC_BYTES = STEPS * STEP_BYTES;
constexpr u32 RAW_BYTES = 4u << 20;
constexpr u32 W_BYTES = (u32)KD * NLOC * 2;
constexpr size_t XC_OFF = 0;
constexpr size_t RAW_OFF = 4u << 20;
constexpr size_t XF_OFF = 8u << 20;
constexpr size_t AG_OFF = XF_OFF + 65536;
constexpr size_t CNT_OFF = AG_OFF + 32768;
constexpr size_t TMO_OFF = CNT_OFF + 16384;
constexpr size_t DBG_OFF = TMO_OFF + 4096;
constexpr size_t WS_NEED = DBG_OFF + 32768;
static_assert(XC_BYTES <= RAW_OFF, "layout");
constexpr u64 DEADLINE = 40000000ull;
constexpr u32 OOB = 0x7fff0000u;

constexpr int NWS = 4;
constexpr u32 L_X = 0;
constexpr u32 L_W = 65536;
constexpr u32 L_JUNK = L_W + NWS * 16384;
constexpr u32 L_LIST = L_JUNK + 256;
constexpr u32 L_PFM = L_LIST + 512;
constexpr u32 L_RED = L_PFM + 32;
constexpr u32 L_TOTAL = L_RED + 64;
static_assert(CFG_D + 2 <= NWS && CFG_D + 1 <= 4, "ring sizes");

struct Params {
  char* ws[NR];
  const char* x;
  const char* w;
  char* out;
  int rank;
};

__device__ __forceinline__ u32 rfl(u32 v) { return (u32)__builtin_amdgcn_readfirstlane((int)v); }
__device__ __forceinline__ u64 rfl64(u64 v) { return ((u64)rfl((u32)(v >> 32)) << 32) | rfl((u32)v); }
__device__ __forceinline__ u32x4 make_srd(const void* p, u32 bytes) {
  u64 a = rfl64((u64)p);
  u32x4 r = {(u32)a, (u32)(a >> 32) & 0xffffu, bytes, 0x00020000u};
  return r;
}
template <int CTRL, int RMASK>
__device__ __forceinline__ u32 dpp0(u32 v) {
  return (u32)__builtin_amdgcn_update_dpp(0, (int)v, CTRL, RMASK, 0xf, false);
}
__device__ __forceinline__ u32 wave_sum(u32 v) {
  v += dpp0<0x111, 0xf>(v); v += dpp0<0x112, 0xf>(v); v += dpp0<0x114, 0xf>(v); v += dpp0<0x118, 0xf>(v);
  v += dpp0<0x142, 0xa>(v); v += dpp0<0x143, 0xc>(v);
  return (u32)__builtin_amdgcn_readlane((int)v, 63);
}
__device__ __forceinline__ u32 umax(u32 a, u32 b) { return a > b ? a : b; }
__device__ __forceinline__ u32 wave_umax(u32 v) {
  v = umax(v, dpp0<0x111, 0xf>(v)); v = umax(v, dpp0<0x112, 0xf>(v));
  v = umax(v, dpp0<0x114, 0xf>(v)); v = umax(v, dpp0<0x118, 0xf>(v));
  v = umax(v, dpp0<0x142, 0xa>(v)); v = umax(v, dpp0<0x143, 0xc>(v));
  return (u32)__builtin_amdgcn_readlane((int)v, 63);
}
__device__ __forceinline__ void wg_barrier() {
  asm volatile("s_waitcnt lgkmcnt(0)\n\ts_barrier" ::: "memory");
}
__device__ __forceinline__ u64 now() { return __builtin_amdgcn_s_memrealtime(); }

#define STAMP(i) do { } while (0)

#define AG_CLOB \
  "a0","a1","a2","a3","a4","a5","a6","a7","a8","a9","a10","a11","a12","a13","a14","a15", \
  "a16","a17","a18","a19","a20","a21","a22","a23","a24","a25","a26","a27","a28","a29","a30","a31", \
  "a32","a33","a34","a35","a36","a37","a38","a39","a40","a41","a42","a43","a44","a45","a46","a47", \
  "a48","a49","a50","a51","a52","a53","a54","a55","a56","a57","a58","a59","a60","a61","a62","a63", \
  "a64","a65","a66","a67","a68","a69","a70","a71","a72","a73","a74","a75","a76","a77","a78","a79", \
  "a80","a81","a82","a83","a84","a85","a86","a87","a88","a89","a90","a91","a92","a93","a94","a95"

template <int SL>
__device__ __forceinline__ void issue_x(u32 vl, u32 vc, u32 vp, u32x4 srd, u32 so) {
  asm volatile(
      "s_nop 4\n\t"
      "buffer_load_dwordx4 a[%c5:%c5+3], %0, %3, %4 offen\n\t"
      "buffer_load_dwordx4 a[%c5+4:%c5+7], %0, %3, %4 offen offset:1024\n\t"
      "buffer_load_dwordx3 a[%c5+16:%c5+18], %1, %3, %4 offen\n\t"
      "buffer_load_dwordx4 a[%c5+8:%c5+11], %0, %3, %4 offen offset:2048\n\t"
      "buffer_load_dwordx4 a[%c5+12:%c5+15], %0, %3, %4 offen offset:3072\n\t"
      "buffer_load_dwordx3 a[%c5+20:%c5+22], %1, %3, %4 offen offset:768\n\t"
      "buffer_load_dword a[%c5+19], %2, %3, %4 offen"
      :: "v"(vl), "v"(vc), "v"(vp), "s"(srd), "s"(so), "i"(SL * 24) : AG_CLOB, "memory");
}
__device__ __forceinline__ void issue_w(u32 v0, u32 v1, u32 v2, u32 v3, u32x4 srd, u32 so, u32 lds) {
  asm volatile(
      "s_mov_b32 m0, %6\n\t"
      "s_nop 3\n\t"
      "buffer_load_dwordx4 %0, %4, %5 offen lds\n\t"
      "buffer_load_dwordx4 %1, %4, %5 offen offset:1024 lds\n\t"
      "buffer_load_dwordx4 %2, %4, %5 offen offset:2048 lds\n\t"
      "buffer_load_dwordx4 %3, %4, %5 offen offset:3072 lds"
      :: "v"(v0), "v"(v1), "v"(v2), "v"(v3), "s"(srd), "s"(so), "s"(lds) : "memory");
}
__device__ __forceinline__ void issue_pf(u32 voff, u32x4 srd, u32 so, u32 lds) {
  asm volatile(
      "s_mov_b32 m0, %3\n\t"
      "s_nop 3\n\t"
      "buffer_load_dword %0, %1, %2 offen lds"
      :: "v"(voff), "s"(srd), "s"(so), "s"(lds) : "memory");
}
template <int N>
__device__ __forceinline__ void wait_vm() { asm volatile("s_waitcnt vmcnt(%c0)" :: "i"(N) : "memory"); }

template <int LB, int CB>
__device__ __forceinline__ void fetch_mb(u32 (&L)[8], u32 (&C)[3]) {
  asm volatile(
      "v_accvgpr_read_b32 %8, a[%c12]\n\t"
      "v_accvgpr_read_b32 %9, a[%c12+1]\n\t"
      "v_accvgpr_read_b32 %10, a[%c12+2]\n\t"
      "v_accvgpr_read_b32 %0, a[%c11]\n\t"
      "v_accvgpr_read_b32 %1, a[%c11+1]\n\t"
      "v_accvgpr_read_b32 %2, a[%c11+2]\n\t"
      "v_accvgpr_read_b32 %3, a[%c11+3]\n\t"
      "v_accvgpr_read_b32 %4, a[%c11+4]\n\t"
      "v_accvgpr_read_b32 %5, a[%c11+5]\n\t"
      "v_accvgpr_read_b32 %6, a[%c11+6]\n\t"
      "v_accvgpr_read_b32 %7, a[%c11+7]"
      : "=v"(L[0]), "=v"(L[1]), "=v"(L[2]), "=v"(L[3]), "=v"(L[4]), "=v"(L[5]), "=v"(L[6]), "=v"(L[7]),
        "=v"(C[0]), "=v"(C[1]), "=v"(C[2])
      : "i"(LB), "i"(CB));
}
template <int R>
__device__ __forceinline__ u32 fetch_a() {
  u32 v;
  asm volatile("v_accvgpr_read_b32 %0, a[%c1]" : "=v"(v) : "i"(R));
  return v;
}

__device__ __forceinline__ void unpack_mb(const u32 (&L)[8], const u32 (&C)[3], u32 eb4, LDSP char* dst) {
  const u32 m7 = 0x07070707u;
  u32 E[8];
  E[0] = (C[0] & m7) + eb4;
  E[1] = ((C[0] >> 3) & m7) + eb4;
  E[2] = (C[1] & m7) + eb4;
  E[3] = ((C[1] >> 3) & m7) + eb4;
  E[4] = (C[2] & m7) + eb4;
  E[5] = ((C[2] >> 3) & m7) + eb4;
  E[6] = (((C[0] >> 6) & 0x03030303u) | ((C[2] >> 4) & 0x04040404u)) + eb4;
  E[7] = (((C[1] >> 6) & 0x03030303u) | ((C[2] >> 5) & 0x04040404u)) + eb4;
#pragma unroll
  for (int s = 0; s < 4; ++s) {
    u32x4 o;
#pragma unroll
    for (int e = 0; e < 2; ++e) {
      const int q = 2 * s + e;
      u32 t0 = __builtin_amdgcn_perm(E[q], L[q], 0x05010400u);
      u32 t1 = __builtin_amdgcn_perm(E[q], L[q], 0x07030602u);
      o[2 * e] = __builtin_amdgcn_alignbit(t0, t0, 1);
      o[2 * e + 1] = __builtin_amdgcn_alignbit(t1, t1, 1);
    }
    *(LDSP u32x4*)(dst + s * 1024) = o;
  }
}

struct Helper {
  u32 vl, vc, vp_all, vp_prim, vw0, vw1, vw2, vw3;
  u32x4 srdx, srdw, srdr;
  u32 meta;
  u32 lane, wb, w;
  u32 pbase;
  u32 pat;
  LDSP char* xl;
};

template <int U>
__device__ __forceinline__ void helper_step(const Helper& h, int j) {
  constexpr int D = CFG_D;
  const int jn = j + D;
  const bool live = jn < STEPS;
  const u32 mn = (u32)__builtin_amdgcn_readlane((int)h.meta, jn & 63);
  u32x4 sx = h.srdx, sw = h.srdw;
  sx[2] = live ? sx[2] : 0u;
  sw[2] = live ? sw[2] : 0u;
  const u32 vp = (mn & 0x10000u) ? h.vp_all : h.vp_prim;
  issue_x<(U + D) & 3>(h.vl, h.vc, vp, sx, (u32)(jn & 63) * STEP_BYTES);
  issue_w(h.vw0, h.vw1, h.vw2, h.vw3, sw, (u32)(jn & 63) * (64u * 4096u),
          L_W + ((U + D) & 3) * 16384u + h.w * 4096u);
  const u32 m = (u32)__builtin_amdgcn_readlane((int)h.meta, j);
  wait_vm<11 * D>();
  LDSP char* dst = h.xl + (U & 1) * 32768;
  {
    u32 L[8], C[3];
    fetch_mb<U * 24, U * 24 + 16>(L, C);
    unpack_mb(L, C, (m & 0xffu) * 0x01010101u, dst);
  }
  {
    u32 L[8], C[3];
    fetch_mb<U * 24 + 8, U * 24 + 20>(L, C);
    unpack_mb(L, C, ((m >> 8) & 0xffu) * 0x01010101u, dst + 4096);
  }
  {
    const u32 e = fetch_a<U * 24 + 19>();
    const u32 ex = e ^ h.pat;
    const u32 a0 = h.pbase + ((ex & 0x3ffu) << 2);
    const u32 a1 = h.pbase + (((ex >> 16) & 0x3ffu) << 2);
    const u32 x0 = (e & 0x7c00u) << ((e >> 11) & 16u);
    const u32 x1 = ((e >> 16) & 0x7c00u) << ((e >> 27) & 16u);
    asm volatile("ds_xor_b32 %0, %1 offset:%c4\n\tds_xor_b32 %2, %3 offset:%c4"
                 :: "v"(a0), "v"(x0), "v"(a1), "v"(x1), "i"((U & 1) * 32768) : "memory");
  }
  if (__builtin_expect((m & 0x60000u) != 0, 0)) {
#pragma unroll 1
    for (int mb = 0; mb < 2; ++mb) {
      if (!((m >> (17 + mb)) & 1)) continue;
      u32x4 r0, r1, r2, r3;
      const u32 so = (((u32)j * 8 + h.wb) * 2 + mb) * 4096u;
      const u32 vo = h.lane * 16;
      asm volatile(
          "s_nop 4\n\t"
          "buffer_load_dwordx4 %0, %4, %5, %6 offen\n\t"
          "buffer_load_dwordx4 %1, %4, %5, %6 offen offset:1024\n\t"
          "buffer_load_dwordx4 %2, %4, %5, %6 offen offset:2048\n\t"
          "buffer_load_dwordx4 %3, %4, %5, %6 offen offset:3072\n\t"
          "s_waitcnt vmcnt(0)"
          : "=&v"(r0), "=&v"(r1), "=&v"(r2), "=&v"(r3) : "v"(vo), "s"(h.srdr), "s"(so) : "memory");
      LDSP char* d2 = dst + mb * 4096;
      *(LDSP u32x4*)(d2) = r0;
      *(LDSP u32x4*)(d2 + 1024) = r1;
      *(LDSP u32x4*)(d2 + 2048) = r2;
      *(LDSP u32x4*)(d2 + 3072) = r3;
    }
  }
  wg_barrier();
}

struct Frags { bf16x8 a[4]; bf16x8 b[2]; };

template <int WOFF, int XOFF>
__device__ __forceinline__ void load_frags(Frags& f, LDSP char* const (&wa)[4], LDSP char* xa) {
#pragma unroll
  for (int mb = 0; mb < 2; ++mb) f.b[mb] = *(LDSP bf16x8*)(xa + XOFF + mb * 4096);
#pragma unroll
  for (int nb = 0; nb < 4; ++nb) {
    bf16x4 lo = __builtin_amdgcn_ds_read_tr16_b64_v4bf16((LDSP bf16x4*)(wa[nb] + WOFF));
    bf16x4 hi = __builtin_amdgcn_ds_read_tr16_b64_v4bf16((LDSP bf16x4*)(wa[nb] + WOFF + 1024));
    f.a[nb] = __builtin_shufflevector(lo, hi, 0, 1, 2, 3, 4, 5, 6, 7);
  }
}
__device__ __forceinline__ void mma(f32x16 (&acc)[4][2], const Frags& f) {
#pragma unroll
  for (int nb = 0; nb < 4; ++nb)
#pragma unroll
    for (int mb = 0; mb < 2; ++mb)
      acc[nb][mb] = __builtin_amdgcn_mfma_f32_32x32x16_bf16(f.a[nb], f.b[mb], acc[nb][mb], 0, 0, 0);
}

template <int U>
__device__ __forceinline__ void mfma_step(f32x16 (&acc)[4][2], Frags& f0, Frags& f1, LDSP char* const (&wa)[4],
                                          LDSP char* xa, bool last, u32 vpf, u32x4 srdx, u64& pfm, u32& pfso) {
  constexpr int W0 = U * 16384, X0 = (U & 1) * 32768;
  constexpr int W1 = ((U + 1) & 3) * 16384, X1 = ((U + 1) & 1) * 32768;
  {
    u32x4 s = srdx;
    s[2] = (pfm & 1) ? s[2] : 0u;
    issue_pf(vpf, s, pfso, L_JUNK);
    pfm >>= 1;
    pfso += STEP_BYTES;
  }
  load_frags<W0 + 4096, X0 + 1024>(f1, wa, xa);
  __builtin_amdgcn_sched_barrier(0);
  mma(acc, f0);
  __builtin_amdgcn_sched_barrier(0);
  load_frags<W0 + 8192, X0 + 2048>(f0, wa, xa);
  __builtin_amdgcn_sched_barrier(0);
  mma(acc, f1);
  __builtin_amdgcn_sched_barrier(0);
  load_frags<W0 + 12288, X0 + 3072>(f1, wa, xa);
  __builtin_amdgcn_sched_barrier(0);
  mma(acc, f0);
  __builtin_amdgcn_sched_barrier(0);
  if (!last) {
    wg_barrier();
    load_frags<W1, X1>(f0, wa, xa);
  }
  __builtin_amdgcn_sched_barrier(0);
  mma(acc, f1);
  __builtin_amdgcn_sched_barrier(0);
}

__global__ void __launch_bounds__(512, 2) ag_gemm_q(Params p) {
  __shared__ __attribute__((aligned(1024))) char smem_raw[L_TOTAL];
  LDSP char* smem = (LDSP char*)smem_raw;
  const u32 tid = threadIdx.x, lane = tid & 63;
  const u32 wave = rfl(tid >> 6);
  const u32 b = blockIdx.x;
  const u32 rank = (u32)p.rank;
  const u32 g = b & 7, hh = (b >> 3) & 1, c = b >> 4;
  char* const myws = (char*)rfl64((u64)p.ws[rank & 7]);
  const char* const srcws = (const char*)rfl64((u64)p.ws[g]);
  const u64 t0 = now();
  GLBP u32* const cnt_ptr = (GLBP u32*)(myws + CNT_OFF + b * 64);
  GLBP u32* const tmo_ptr = (GLBP u32*)(myws + TMO_OFF);
  LDSP u32* red = (LDSP u32*)(smem + L_RED);

  if (wave < 4) {
    const u32 w = wave;
    STAMP(0);
    const u32 est = b >> 2, mbi = (b & 3) * 4 + w, ewb = mbi >> 1, mb = mbi & 1;
    u32x4 xv[4];
    {
      const char* xp = p.x + (size_t)(mbi * 32 + (lane & 31)) * (KD * 2) + est * 128 + (lane >> 5) * 16;
#pragma unroll
      for (int s = 0; s < 4; ++s) xv[s] = *(const GLBP u32x4*)(xp + s * 32);
    }
    const u32 cntv = __hip_atomic_load(cnt_ptr, __ATOMIC_RELAXED, __HIP_MEMORY_SCOPE_SYSTEM);
    char* const peer = p.ws[lane & 7];
    LDSP u32* list = (LDSP u32*)(smem + L_LIST + w * 128);
    if (lane < 32) list[lane] = 0;
    if (tid < 16) red[tid] = 0;
    wg_barrier();
    const u32 seq = rfl(cntv) + 1;
    {
      u32 L[8], E[8];
      u32 sum = 0;
#pragma unroll
      for (int q = 0; q < 8; ++q) {
        const u32 x0 = xv[q >> 1][2 * (q & 1)], x1 = xv[q >> 1][2 * (q & 1) + 1];
        const u32 ta = __builtin_amdgcn_alignbit(x0, x0, 31), tb = __builtin_amdgcn_alignbit(x1, x1, 31);
        L[q] = __builtin_amdgcn_perm(tb, ta, 0x06040200u);
        E[q] = __builtin_amdgcn_perm(tb, ta, 0x07050301u);
        sum = __builtin_amdgcn_sad_u8(E[q], 0, sum);
      }
      __amdgpu_buffer_rsrc_t rc = __builtin_amdgcn_make_buffer_rsrc(myws + XC_OFF, 0, (int)XC_BYTES, 0x00020000);
      const u32 blk = est * STEP_BYTES + ewb * WB_BYTES;
      {
        u32x4 l0 = {L[0], L[1], L[2], L[3]}, l1 = {L[4], L[5], L[6], L[7]};
        __builtin_amdgcn_raw_buffer_store_b128(l0, rc, lane * 16, blk + mb * 2048, 17);
        __builtin_amdgcn_raw_buffer_store_b128(l1, rc, lane * 16 + 1024, blk + mb * 2048, 17);
      }
      int ebi = (int)((wave_sum(sum) + 3727u + 2047u) >> 11) - 7;
      ebi = ebi < 0 ? 0 : (ebi > 248 ? 248 : ebi);
      const u32 eb = (u32)ebi;
      const u32 eb4 = eb * 0x01010101u, kk = (8u - (eb & 7u)) * 0x01010101u, m7 = 0x07070707u;
      u32 cd[8], X[8];
#pragma unroll
      for (int q = 0; q < 8; ++q) {
        cd[q] = ((E[q] & m7) + kk) & m7;
        X[q] = E[q] ^ (cd[q] + eb4);
      }
      {
        u32x3 cw;
        cw[0] = cd[0] | (cd[1] << 3) | ((cd[6] & 0x03030303u) << 6);
        cw[1] = cd[2] | (cd[3] << 3) | ((cd[7] & 0x03030303u) << 6);
        cw[2] = cd[4] | (cd[5] << 3) | ((cd[6] & 0x04040404u) << 4) | ((cd[7] & 0x04040404u) << 5);
        __builtin_amdgcn_raw_buffer_store_b96(cw, rc, lane * 12, blk + 4096 + mb * 768, 17);
      }
      u32 cnt = 0;
      LDSP unsigned short* list16 = (LDSP unsigned short*)list;
#pragma unroll
      for (int q = 0; q < 8; ++q) {
        u32 xq = X[q];
        while (cnt <= 64) {
          const u64 mk = __builtin_amdgcn_ballot_w64(xq != 0);
          if (mk == 0) break;
          if (xq != 0) {
            const u32 by = (u32)__builtin_ctz(xq) >> 3;
            const u32 pay = (xq >> (8 * by + 3)) & 31u;
            const u32 slot = cnt + __builtin_amdgcn_mbcnt_hi((u32)(mk >> 32), __builtin_amdgcn_mbcnt_lo((u32)mk, 0));
            const u32 idx = (q >> 1) * 256 + lane * 4 + 2 * (q & 1) + (by >> 1);
            const u32 e = ((by & 1) << 15) | (pay << 10) | (idx ^ (mb * 16 + ((slot >> 1) & 15)));
            if (slot < 64) list16[slot] = (unsigned short)e;
            xq &= ~(0xffu << (8 * by));
          }
          cnt += (u32)__builtin_popcountll(mk);
        }
      }
      const bool raw = CFG_FORCE_RAW || cnt > 64;
      if (lane < 32) {
        const u32 lv = list[lane];
        __builtin_amdgcn_raw_buffer_store_b32(lv, rc, (lane >> 4) * 128 + (lane & 15) * 4, blk + 5632 + mb * 64, 17);
      }
      if (raw) {
        __amdgpu_buffer_rsrc_t rr = __builtin_amdgcn_make_buffer_rsrc(myws + RAW_OFF, 0, (int)RAW_BYTES, 0x00020000);
#pragma unroll
        for (int s = 0; s < 4; ++s)
          __builtin_amdgcn_raw_buffer_store_b128(xv[s], rr, lane * 16 + s * 1024, ((est * 8 + ewb) * 2 + mb) * 4096, 17);
      }
      asm volatile("s_waitcnt vmcnt(0)" ::: "memory");
      if (lane < NR) {
        u32x2 gr = {seq, eb | (cnt > 32 ? 0x100u : 0u) | (raw ? 0x200u : 0u)};
        char* fp = peer + XF_OFF + ((rank * 8 + ewb) * 64 + est) * 16 + mb * 8;
        asm volatile("global_store_dwordx2 %0, %1, off sc0 sc1" :: "v"(fp), "v"(gr) : "memory");
      }
    }
    STAMP(1);
    f32x16 acc[4][2];
#pragma unroll
    for (int nb = 0; nb < 4; ++nb)
#pragma unroll
      for (int m2 = 0; m2 < 2; ++m2)
#pragma unroll
        for (int r = 0; r < 16; ++r) acc[nb][m2][r] = 0.f;
    LDSP char* wa[4];
    {
      const u32 h = lane >> 5, gq = (lane >> 4) & 1, q = (lane >> 2) & 3, pp = lane & 3;
#pragma unroll
      for (int nb = 0; nb < 4; ++nb)
        wa[nb] = smem + L_W + (8 * h + q) * 256 + ((nb ^ q) * 64) + 32 * (pp & 1) + 16 * gq + 8 * (pp >> 1);
    }
    LDSP char* xa = smem + L_X + w * 8192 + lane * 16;
    const u32x4 srdx = make_srd(srcws + XC_OFF, XC_BYTES);
    const u32 line = 3 * c + lane;
    const u32 vpf = (lane < 3 && line < 46) ? (4 * hh + w) * WB_BYTES + line * 128 : OOB;
    wg_barrier();
    STAMP(2);
    u64 pfm;
    {
      const u32x2 pm = *(LDSP u32x2*)(smem + L_PFM + w * 8);
      pfm = (((u64)rfl(pm[1]) << 32) | rfl(pm[0])) >> CFG_P;
    }
    u32 pfso = CFG_P * STEP_BYTES;
    Frags f0, f1;
    load_frags<0, 0>(f0, wa, xa);
#pragma unroll 1
    for (int i = 0; i < STEPS; i += 4) {
      mfma_step<0>(acc, f0, f1, wa, xa, false, vpf, srdx, pfm, pfso);
      mfma_step<1>(acc, f0, f1, wa, xa, false, vpf, srdx, pfm, pfso);
      mfma_step<2>(acc, f0, f1, wa, xa, false, vpf, srdx, pfm, pfso);
      mfma_step<3>(acc, f0, f1, wa, xa, i == STEPS - 4, vpf, srdx, pfm, pfso);
    }
    STAMP(3);
    u32 am = 0;
#pragma unroll
    for (int nb = 0; nb < 4; ++nb)
#pragma unroll
      for (int m2 = 0; m2 < 2; ++m2) {
        asm volatile("" : "+a"(acc[nb][m2]));
#pragma unroll
        for (int r = 0; r < 16; r += 2) {
          const float e0 = acc[nb][m2][r], e1 = acc[nb][m2][r + 1];
          asm("v_max3_f32 %0, |%1|, |%2|, %0" : "+v"(am) : "v"(e0), "v"(e1));
        }
        asm volatile("" : "+v"(am));
      }
    am = wave_umax(am);
    u32 ticket = 0;
    if (lane == 0) {
      __hip_atomic_fetch_max(&red[0], am, __ATOMIC_RELAXED, __HIP_MEMORY_SCOPE_WORKGROUP);
      ticket = __hip_atomic_fetch_add(&red[1], 1u, __ATOMIC_RELAXED, __HIP_MEMORY_SCOPE_WORKGROUP);
    }
    if (rfl(ticket) == 3) {
      u32 wm = __hip_atomic_load(&red[0], __ATOMIC_RELAXED, __HIP_MEMORY_SCOPE_WORKGROUP);
      if (__hip_atomic_load(&red[2], __ATOMIC_RELAXED, __HIP_MEMORY_SCOPE_WORKGROUP)) wm = 0x7fffffffu;
      if (lane < NR) {
        u32x2 gr = {seq, wm};
        char* fp = peer + AG_OFF + (((seq & 1) * 8 + rank) * 256 + b) * 8;
        asm volatile("global_store_dwordx2 %0, %1, off sc0 sc1" :: "v"(fp), "v"(gr) : "memory");
      }
    }
    STAMP(4);
    wg_barrier();
    STAMP(5);
    const u32x4 sw = *(LDSP u32x4*)(red + 4);
    const float gmax = __builtin_bit_cast(float, umax(umax(sw[0], sw[1]), umax(sw[2], sw[3])));
    const float inv = 448.0f / gmax;
    LDSP char* tr = smem + L_X + w * 8192;
    {
      const u32 r = lane & 31, h = lane >> 5;
#pragma unroll
      for (int m2 = 0; m2 < 2; ++m2)
#pragma unroll
        for (int nb = 0; nb < 4; ++nb) {
          u32x4 o;
          asm volatile("" : "+a"(acc[nb][m2]));
#pragma unroll
          for (int d = 0; d < 4; ++d) {
            int v = __builtin_amdgcn_cvt_pk_fp8_f32(acc[nb][m2][4 * d] * inv, acc[nb][m2][4 * d + 1] * inv, 0, false);
            v = __builtin_amdgcn_cvt_pk_fp8_f32(acc[nb][m2][4 * d + 2] * inv, acc[nb][m2][4 * d + 3] * inv, v, true);
            o[d] = (u32)v;
          }
          *(LDSP u32x4*)(tr + (m2 * 32 + r) * 128 + (((2 * nb + h) ^ (r & 7)) * 16)) = o;
        }
    }
    {
      const u32 rr = lane >> 3, ch = lane & 7;
      char* op = p.out + (size_t)(g * MLOC + hh * 256 + w * 64 + rr) * NLOC + c * 128 + ch * 16;
#pragma unroll
      for (int i = 0; i < 8; ++i) {
        const u32x4 o = *(LDSP u32x4*)(tr + (i * 8 + rr) * 128 + ((ch ^ rr) * 16));
        *(GLBP u32x4*)(op + (size_t)i * 8 * NLOC) = o;
      }
    }
    if (tid == 0) __hip_atomic_store(cnt_ptr, seq, __ATOMIC_RELAXED, __HIP_MEMORY_SCOPE_SYSTEM);
    asm volatile("s_waitcnt vmcnt(0)" ::: "memory");
    STAMP(6);
  } else {
    Helper h;
    const u32 w = wave - 4;
    h.w = w; h.lane = lane; h.wb = 4 * hh + w;
    h.srdw = make_srd(p.w, W_BYTES);
    {
      const u32 rq = lane >> 4;
      const u32 base = (16 * w + rq) * 4096 + c * 256 + (((lane & 15) ^ (rq << 2)) * 16);
      h.vw0 = base; h.vw1 = base + 16384 - 1024; h.vw2 = base + 2 * (16384 - 1024); h.vw3 = base + 3 * (16384 - 1024);
    }
    if (w == 0) {
      asm volatile("buffer_inv sc0 sc1\n\ts_waitcnt vmcnt(0)" ::: "memory");
    }
#pragma unroll
    for (int d = 0; d < CFG_D; ++d)
      issue_w(h.vw0, h.vw1, h.vw2, h.vw3, h.srdw, (u32)d * (64u * 4096u), L_W + d * 16384u + w * 4096u);
    wg_barrier();
    u32 cntv;
    asm volatile("global_load_dword %0, %1, off sc0 sc1\n\ts_waitcnt vmcnt(0)" : "=&v"(cntv) : "v"(cnt_ptr) : "memory");
    const u32 seq = rfl(cntv) + 1;
    bool ok = true;
    {
      const char* fp = myws + XF_OFF + ((g * 8 + h.wb) * 64 + lane) * 16;
      u32x4 f;
      for (;;) {
        u32 tm;
        asm volatile("global_load_dwordx4 %0, %2, off sc0 sc1\n\tglobal_load_dword %1, %3, off sc1\n\ts_waitcnt vmcnt(0)"
                     : "=&v"(f), "=&v"(tm) : "v"(fp), "v"(tmo_ptr) : "memory");
        if (__builtin_amdgcn_ballot_w64(f[0] < seq || f[2] < seq) == 0) break;
        if (rfl(tm) == seq || now() - t0 > DEADLINE) {
          if (lane == 0) {
            __hip_atomic_store(tmo_ptr, seq, __ATOMIC_RELAXED, __HIP_MEMORY_SCOPE_AGENT);
            __hip_atomic_store(&red[2], 1u, __ATOMIC_RELAXED, __HIP_MEMORY_SCOPE_WORKGROUP);
          }
          ok = false;
          break;
        }
        __builtin_amdgcn_s_sleep(2);
      }
      h.meta = ok ? ((f[1] & 0xffu) | ((f[3] & 0xffu) << 8) | (((f[1] | f[3]) & 0x100u) << 8) |
                     ((f[1] & 0x200u) << 8) | ((f[3] & 0x200u) << 9)) : 0u;
    }
    STAMP(8);
    h.srdx = make_srd(srcws + XC_OFF, ok ? XC_BYTES : 0u);
    h.srdr = make_srd(srcws + RAW_OFF, ok ? RAW_BYTES : 0u);
    const u32 wbo = h.wb * WB_BYTES;
    h.vl = wbo + lane * 16;
    h.vc = wbo + 4096 + lane * 12;
    h.vp_all = wbo + 5632 + lane * 4;
    h.vp_prim = lane < 32 ? h.vp_all : OOB;
    h.xl = smem + L_X + w * 8192 + lane * 16;
    h.pbase = L_X + w * 8192 + ((lane >> 4) & 1) * 4096;
    h.pat = (lane & 31) * 0x00010001u;
    {
      const u64 ovm = __builtin_amdgcn_ballot_w64((h.meta & 0x10000u) != 0);
      const u64 pfm = ok ? (c == 15 ? ovm : ~0ull) : 0ull;
      const u32 st = lane / 3, line = 3 * c + lane % 3;
      const u32 v = (st < CFG_P && line < 46 && ((pfm >> st) & 1)) ? st * STEP_BYTES + wbo + line * 128 : OOB;
      issue_pf(v, h.srdx, 0, L_JUNK);
      if (lane == 0) { u32x2 pm = {(u32)pfm, (u32)(pfm >> 32)}; *(LDSP u32x2*)(smem + L_PFM + w * 8) = pm; }
    }
    {
      const u32 m0 = (u32)__builtin_amdgcn_readlane((int)h.meta, 0);
      issue_x<0>(h.vl, h.vc, (m0 & 0x10000u) ? h.vp_all : h.vp_prim, h.srdx, 0);
      issue_w(h.vw0, h.vw1, h.vw2, h.vw3, h.srdw, 0, L_W + w * 4096u);
      if (CFG_D > 1) {
        const u32 m1 = (u32)__builtin_amdgcn_readlane((int)h.meta, 1);
        issue_x<1>(h.vl, h.vc, (m1 & 0x10000u) ? h.vp_all : h.vp_prim, h.srdx, STEP_BYTES);
        issue_w(h.vw0, h.vw1, h.vw2, h.vw3, h.srdw, 64u * 4096u, L_W + 16384u + w * 4096u);
      }
    }
#pragma unroll 1
    for (int j = 0; j < STEPS; j += 4) {
      helper_step<0>(h, j);
      helper_step<1>(h, j + 1);
      helper_step<2>(h, j + 2);
      helper_step<3>(h, j + 3);
    }
    STAMP(9);
    u32 gm = 0;
    {
      const char* gp = myws + AG_OFF + (seq & 1) * 16384 + w * 4096 + lane * 16;
      bool done = false;
      for (;;) {
        u32 tm = 0;
        if (!done) {
          u32x4 a0, a1, a2, a3;
          asm volatile("global_load_dwordx4 %0, %5, off sc0 sc1\n\tglobal_load_dwordx4 %1, %5, off offset:1024 sc0 sc1\n\t"
                       "global_load_dwordx4 %2, %5, off offset:2048 sc0 sc1\n\tglobal_load_dwordx4 %3, %5, off offset:3072 sc0 sc1\n\t"
                       "global_load_dword %4, %6, off sc1\n\ts_waitcnt vmcnt(0)"
                       : "=&v"(a0), "=&v"(a1), "=&v"(a2), "=&v"(a3), "=&v"(tm) : "v"(gp), "v"(tmo_ptr) : "memory");
          done = a0[0] >= seq && a0[2] >= seq && a1[0] >= seq && a1[2] >= seq &&
                 a2[0] >= seq && a2[2] >= seq && a3[0] >= seq && a3[2] >= seq;
          gm = umax(umax(umax(a0[1], a0[3]), umax(a1[1], a1[3])), umax(umax(a2[1], a2[3]), umax(a3[1], a3[3])));
        }
        if (__builtin_amdgcn_ballot_w64(!done) == 0) break;
        if (__builtin_amdgcn_ballot_w64(tm == seq) != 0 || now() - t0 > DEADLINE) {
          if (lane == 0) __hip_atomic_store(tmo_ptr, seq, __ATOMIC_RELAXED, __HIP_MEMORY_SCOPE_AGENT);
          gm = 0x7fffffffu;
          break;
        }
        __builtin_amdgcn_s_sleep(8);
      }
    }
    gm = wave_umax(gm);
    if (lane == 0) red[4 + w] = gm;
    STAMP(10);
    wg_barrier();
  }
}

}

extern "C" void kernel_launch(int rank, int nranks, ncclComm_t comm, hipStream_t stream,
                              void* const* d_in, const int* in_sizes, int n_in,
                              void* d_out, int out_size, const int* in_dtypes, int out_dtype,
                              void* const* d_out_all, void* const* d_ws_all, size_t ws_bytes) {
  (void)comm; (void)in_dtypes; (void)out_dtype; (void)d_out_all;
  if (nranks != NR || (unsigned)rank >= (unsigned)NR || n_in < 2 || ws_bytes < WS_NEED ||
      in_sizes[0] != MLOC * KD || in_sizes[1] != KD * NLOC || out_size != NR * MLOC * NLOC) {
    fprintf(stderr, "unexpected problem shape\n");
    return;
  }
  Params p;
  for (int i = 0; i < NR; ++i) p.ws[i] = (char*)d_ws_all[i];
  p.x = (const char*)d_in[0];
  p.w = (const char*)d_in[1];
  p.out = (char*)d_out;
  p.rank = rank;
  hipLaunchKernelGGL(ag_gemm_q, dim3(NWG), dim3(512), 0, stream, p);
  HIP_CHECK(hipGetLastError());
}
```

```cpp
#include <hip/hip_runtime.h>
#include <rccl/rccl.h>
#include <cstdio>
#include <cstdint>

#define HIP_CHECK(expr) do { hipError_t _e = (expr); if (_e != hipSuccess) { \
    fprintf(stderr, "HIP error %s at %s:%d\n", hipGetErrorString(_e), __FILE__, __LINE__); \
    return; } } while (0)

namespace {

typedef unsigned u32;
typedef unsigned long long u64;
typedef u32 u32x2 __attribute__((ext_vector_type(2)));
typedef u32 u32x3 __attribute__((ext_vector_type(3)));
typedef u32 u32x4 __attribute__((ext_vector_type(4)));
typedef u32 u32x8 __attribute__((ext_vector_type(8)));
typedef float f32x16 __attribute__((ext_vector_type(16)));
#define LDSP __attribute__((address_space(3)))
#define GLBP __attribute__((address_space(1)))

#define GROUP_BITS 0xB2
#define CFG_FORCE_RAW 0
#define INV_SLP 2
#define NC 16
#define CFG_NSW 4
#define CFG_SLP2 2

constexpr int NR = 8;
constexpr int KD = 4096, NLOC = 2048, MLOC = 512;
constexpr int STEPS = 64;
constexpr int NWG = 256;
constexpr u32 SELF_XCD = 1;
constexpr u32 WB_BYTES = 5888;
constexpr u32 STEP_BYTES = 8 * WB_BYTES;
constexpr u32 XC_BYTES = STEPS * STEP_BYTES;
constexpr u32 RAW_BYTES = 4u << 20;
constexpr u32 W_BYTES = (u32)KD * NLOC * 2;
constexpr size_t XC_OFF = 65536;
constexpr size_t XF_OFF = XC_OFF + XC_BYTES;
constexpr u32 XF_B = 64 * 8 + 128;
constexpr size_t AGL_OFF = XF_OFF + 128 * XF_B;
constexpr size_t AGR_OFF = AGL_OFF + 8192;
constexpr size_t CNT_OFF = AGR_OFF + 65536;
constexpr size_t INV_OFF = CNT_OFF + 16384;
constexpr size_t TMO_OFF = INV_OFF + 16384;
constexpr size_t DBG_OFF = TMO_OFF + 3072;
constexpr size_t RAW_OFF = 4u << 20;
constexpr size_t WS_NEED = RAW_OFF + RAW_BYTES;
static_assert(DBG_OFF + 65536 <= RAW_OFF, "layout");
static_assert(STEP_BYTES == 47104, "the loop text holds this number");
constexpr u64 DEADLINE = 40000000ull;
constexpr u64 SLOW = 100000ull;

constexpr u32 L_W = 0;
constexpr u32 L_ENC = 6 * 8192;
constexpr u32 L_PL = 131072;
constexpr u32 L_RED = L_PL + 16384;
constexpr u32 L_DUMP = L_RED + 128;
constexpr u32 L_TOTAL = L_DUMP + 256;

struct Params {
  char* ws[NR];
  const char* x;
  const char* w;
  char* out;
  char* myws;
  u32 rank;
  u32 order;
};

__device__ __forceinline__ u32 rfl(u32 v) { return (u32)__builtin_amdgcn_readfirstlane((int)v); }
__device__ __forceinline__ u64 rfl64(u64 v) { return ((u64)rfl((u32)(v >> 32)) << 32) | rfl((u32)v); }
__device__ __forceinline__ u32x4 make_srd(const void* p, u32 bytes) {
  u64 a = rfl64((u64)p);
  u32x4 r = {(u32)a, (u32)(a >> 32) & 0xffffu, bytes, 0x00020000u};
  return r;
}
template <int CTRL, int RMASK>
__device__ __forceinline__ u32 dpp0(u32 v) {
  return (u32)__builtin_amdgcn_update_dpp(0, (int)v, CTRL, RMASK, 0xf, false);
}
__device__ __forceinline__ u32 wave_scan(u32 v) {
  v += dpp0<0x111, 0xf>(v); v += dpp0<0x112, 0xf>(v); v += dpp0<0x114, 0xf>(v); v += dpp0<0x118, 0xf>(v);
  v += dpp0<0x142, 0xa>(v); v += dpp0<0x143, 0xc>(v);
  return v;
}
__device__ __forceinline__ u32 last_lane(u32 v) { return (u32)__builtin_amdgcn_readlane((int)v, 63); }
__device__ __forceinline__ u32 umax(u32 a, u32 b) { return a > b ? a : b; }
__device__ __forceinline__ u32 wave_umax(u32 v) {
  v = umax(v, dpp0<0x111, 0xf>(v)); v = umax(v, dpp0<0x112, 0xf>(v));
  v = umax(v, dpp0<0x114, 0xf>(v)); v = umax(v, dpp0<0x118, 0xf>(v));
  v = umax(v, dpp0<0x142, 0xa>(v)); v = umax(v, dpp0<0x143, 0xc>(v));
  return last_lane(v);
}
__device__ __forceinline__ void wg_barrier() {
  asm volatile("s_waitcnt lgkmcnt(0)\n\ts_barrier" ::: "memory");
}
__device__ __forceinline__ u64 now() { return __builtin_amdgcn_s_memrealtime(); }
__device__ __forceinline__ bool expired(u64 t0, GLBP u32* tmo, u32 seq) {
  const u64 dt = now() - t0;
  if (__builtin_expect(dt < SLOW, 1)) return false;
  if (dt <= DEADLINE && __hip_atomic_load(tmo, __ATOMIC_RELAXED, __HIP_MEMORY_SCOPE_AGENT) != seq) return false;
  __hip_atomic_store(tmo, seq, __ATOMIC_RELAXED, __HIP_MEMORY_SCOPE_AGENT);
  return true;
}

#define STAMP(i) do { } while (0)

#define MAIN_NP 2
#define MAIN_WD 4
#define MAIN_XD 2
#define MAIN_SOX 3
#define MAIN_CLOB "v22","v23","v24","v25","v26","v27","v28","v29","v30","v31","v40","v41","v42","v43","v44","v45","v46","v47","v48","v49","v50","v51","v52","v53","v54","v55","v56","v57","v58","v59","v60","v61","v62","v63","v64","v65","v66","v67","v68","v69","v70","v71","v72","v73","v74","v75","v76","v77","v78","v79","v80","v81","v82","v83","v84","v85","v86","v87","v88","v89","v90","v91","v92","v93","v94","v95","v96","v97","v98","v99","v100","v101","v102","v103","v104","v105","v106","v107","v108","v109","v110","v111","v112","v113","v114","v115","v116","v117","v118","v119","v120","v121","v122","v123","v124","v125","v126","v127","v128","v129","v130","v131","v132","v133","v134","v135","v136","v137","v138","v139","v140","v141","v142","v143","v144","v145","v146","v147","v148","v149","v150","v151","v152","v153","v154","v155","v156","v157","v158","v159","v160","v161","v162","v163","v164","v165","v166","v167","v168","v169","v170","v171","v172","v173","v174","v175","v176","v177","v178","v179","v180","v181","v182","v183","v184","v185","v186","v187","v188","v189","v190","v191","s52","s53","s54","s55","s56","s57","s58","s59","s60","s61"
#define MAIN_ASM \
  "s_nop 4\ns_mov_b32 s58, 0\ns_mov_b32 s59, 524288\ns_bitcmp1_b32 s64, 0\ns_cselect_b32 s60, 0x80, 0\nv_and_or_b32 v23, v17, s60, v16\n" \
  "buffer_load_dwordx3 v[88:90], v15, s[40:43], s58 offen\nbuffer_load_ushort v91, v23, s[40:43], s58 offen\n" \
  "buffer_load_dwordx4 v[80:83], v14, s[40:43], s58 offen\nbuffer_load_dwordx4 v[84:87], v14, s[40:43], s58 offen offset:1024\n" \
  "s_add_u32 s58, s58, 47104\ns_add_u32 m0, s62, 16384\nbuffer_load_dwordx4 v18, s[44:47], s59 offen offset:0 lds\n" \
  "buffer_load_dwordx4 v19, s[44:47], s59 offen offset:1024 lds\ns_add_u32 s59, s59, 262144\ns_bitcmp1_b32 s64, 1\ns_cselect_b32 s60, 0x80, 0\n" \
  "v_and_or_b32 v23, v17, s60, v16\nbuffer_load_dwordx3 v[100:102], v15, s[40:43], s58 offen\nbuffer_load_ushort v103, v23, s[40:43], s58 offen\n" \
  "buffer_load_dwordx4 v[92:95], v14, s[40:43], s58 offen\nbuffer_load_dwordx4 v[96:99], v14, s[40:43], s58 offen offset:1024\n" \
  "s_add_u32 s58, s58, 47104\nv_mov_b32 v40, v21\nv_mov_b32 v41, v21\nv_mov_b32 v42, v21\nv_mov_b32 v43, v21\ns_mov_b32 s52, 0x83838383\n" \
  "s_mov_b32 s53, 0x3030303\ns_mov_b32 s54, 0x80808080\ns_mov_b32 s55, 0x5010400\ns_mov_b32 s56, 0x7030602\ns_mov_b32 s57, 0x7ff\ns_mov_b32 s61, 0x0\n" \
  "ds_write_b128 v20, v[40:43]\nds_write_b128 v20, v[40:43] offset:1024\nv_mov_b32 v128, 0\nv_mov_b32 v129, 0\nv_mov_b32 v130, 0\nv_mov_b32 v131, 0\n" \
  "v_mov_b32 v132, 0\nv_mov_b32 v133, 0\nv_mov_b32 v134, 0\nv_mov_b32 v135, 0\nv_mov_b32 v136, 0\nv_mov_b32 v137, 0\nv_mov_b32 v138, 0\n" \
  "v_mov_b32 v139, 0\nv_mov_b32 v140, 0\nv_mov_b32 v141, 0\nv_mov_b32 v142, 0\nv_mov_b32 v143, 0\nv_mov_b32 v144, 0\nv_mov_b32 v145, 0\n" \
  "v_mov_b32 v146, 0\nv_mov_b32 v147, 0\nv_mov_b32 v148, 0\nv_mov_b32 v149, 0\nv_mov_b32 v150, 0\nv_mov_b32 v151, 0\nv_mov_b32 v152, 0\n" \
  "v_mov_b32 v153, 0\nv_mov_b32 v154, 0\nv_mov_b32 v155, 0\nv_mov_b32 v156, 0\nv_mov_b32 v157, 0\nv_mov_b32 v158, 0\nv_mov_b32 v159, 0\n" \
  "v_mov_b32 v160, 0\nv_mov_b32 v161, 0\nv_mov_b32 v162, 0\nv_mov_b32 v163, 0\nv_mov_b32 v164, 0\nv_mov_b32 v165, 0\nv_mov_b32 v166, 0\n" \
  "v_mov_b32 v167, 0\nv_mov_b32 v168, 0\nv_mov_b32 v169, 0\nv_mov_b32 v170, 0\nv_mov_b32 v171, 0\nv_mov_b32 v172, 0\nv_mov_b32 v173, 0\n" \
  "v_mov_b32 v174, 0\nv_mov_b32 v175, 0\nv_mov_b32 v176, 0\nv_mov_b32 v177, 0\nv_mov_b32 v178, 0\nv_mov_b32 v179, 0\nv_mov_b32 v180, 0\n" \
  "v_mov_b32 v181, 0\nv_mov_b32 v182, 0\nv_mov_b32 v183, 0\nv_mov_b32 v184, 0\nv_mov_b32 v185, 0\nv_mov_b32 v186, 0\nv_mov_b32 v187, 0\n" \
  "v_mov_b32 v188, 0\nv_mov_b32 v189, 0\nv_mov_b32 v190, 0\nv_mov_b32 v191, 0\nv_mov_b32 v192, 0\nv_mov_b32 v193, 0\nv_mov_b32 v194, 0\n" \
  "v_mov_b32 v195, 0\nv_mov_b32 v196, 0\nv_mov_b32 v197, 0\nv_mov_b32 v198, 0\nv_mov_b32 v199, 0\nv_mov_b32 v200, 0\nv_mov_b32 v201, 0\n" \
  "v_mov_b32 v202, 0\nv_mov_b32 v203, 0\nv_mov_b32 v204, 0\nv_mov_b32 v205, 0\nv_mov_b32 v206, 0\nv_mov_b32 v207, 0\nv_mov_b32 v208, 0\n" \
  "v_mov_b32 v209, 0\nv_mov_b32 v210, 0\nv_mov_b32 v211, 0\nv_mov_b32 v212, 0\nv_mov_b32 v213, 0\nv_mov_b32 v214, 0\nv_mov_b32 v215, 0\n" \
  "v_mov_b32 v216, 0\nv_mov_b32 v217, 0\nv_mov_b32 v218, 0\nv_mov_b32 v219, 0\nv_mov_b32 v220, 0\nv_mov_b32 v221, 0\nv_mov_b32 v222, 0\n" \
  "v_mov_b32 v223, 0\nv_mov_b32 v224, 0\nv_mov_b32 v225, 0\nv_mov_b32 v226, 0\nv_mov_b32 v227, 0\nv_mov_b32 v228, 0\nv_mov_b32 v229, 0\n" \
  "v_mov_b32 v230, 0\nv_mov_b32 v231, 0\nv_mov_b32 v232, 0\nv_mov_b32 v233, 0\nv_mov_b32 v234, 0\nv_mov_b32 v235, 0\nv_mov_b32 v236, 0\n" \
  "v_mov_b32 v237, 0\nv_mov_b32 v238, 0\nv_mov_b32 v239, 0\nv_mov_b32 v240, 0\nv_mov_b32 v241, 0\nv_mov_b32 v242, 0\nv_mov_b32 v243, 0\n" \
  "v_mov_b32 v244, 0\nv_mov_b32 v245, 0\nv_mov_b32 v246, 0\nv_mov_b32 v247, 0\nv_mov_b32 v248, 0\nv_mov_b32 v249, 0\nv_mov_b32 v250, 0\n" \
  "v_mov_b32 v251, 0\nv_mov_b32 v252, 0\nv_mov_b32 v253, 0\nv_mov_b32 v254, 0\nv_mov_b32 v255, 0\ns_barrier\nds_read_b64_tr_b16 v[64:65], v32 offset:0\n" \
  "ds_read_b64_tr_b16 v[66:67], v32 offset:1024\nds_read_b64_tr_b16 v[68:69], v33 offset:0\nds_read_b64_tr_b16 v[70:71], v33 offset:1024\n" \
  "ds_read_b64_tr_b16 v[72:73], v34 offset:0\nds_read_b64_tr_b16 v[74:75], v34 offset:1024\ns_waitcnt vmcnt(8)\nv_bfi_b32 v22, s57, v91, v20\n" \
  "v_lshrrev_b32 v31, 9, v91\nds_write_b8 v22, v31\nds_read_b128 v[40:43], v20\nds_read_b128 v[44:47], v20 offset:1024\nds_write_b8 v22, v21\n" \
  "v_alignbit_b32 v25, v88, v88, 3\nv_alignbit_b32 v26, v89, v89, 3\nv_alignbit_b32 v27, v90, v90, 3\nv_lshrrev_b32 v28, 5, v88\n" \
  "v_lshrrev_b32 v29, 5, v89\nv_lshlrev_b32 v30, 2, v90\nv_lshlrev_b32 v31, 1, v90\ns_waitcnt lgkmcnt(1)\nv_bfi_b32 v40, s52, v88, v40\n" \
  "v_bfi_b32 v42, s52, v89, v42\ns_add_u32 m0, s62, 24576\nv_bfi_b32 v44, s52, v90, v44\nbuffer_load_dwordx4 v18, s[44:47], s59 offen offset:0 lds\n" \
  "v_bfi_b32 v41, s52, v25, v41\nv_bfi_b32 v43, s52, v26, v43\nbuffer_load_dwordx4 v19, s[44:47], s59 offen offset:1024 lds\ns_add_u32 s59, s59, 262144\n" \
  "v_bfi_b32 v45, s52, v27, v45\nv_bfi_b32 v46, s53, v28, v46\nv_bfi_b32 v47, s53, v29, v47\nv_bfi_b32 v46, s54, v30, v46\nv_bfi_b32 v47, s54, v31, v47\n" \
  "s_waitcnt vmcnt(8)\nv_perm_b32 v48, v40, v80, s55\nv_perm_b32 v49, v40, v80, s56\nv_perm_b32 v50, v41, v81, s55\nv_perm_b32 v51, v41, v81, s56\n" \
  "v_perm_b32 v52, v42, v82, s55\nv_perm_b32 v53, v42, v82, s56\nv_perm_b32 v54, v43, v83, s55\nv_perm_b32 v55, v43, v83, s56\n" \
  "v_perm_b32 v56, v44, v84, s55\nv_perm_b32 v57, v44, v84, s56\nv_perm_b32 v58, v45, v85, s55\nv_perm_b32 v59, v45, v85, s56\n" \
  "v_perm_b32 v60, v46, v86, s55\nv_perm_b32 v61, v46, v86, s56\nv_perm_b32 v62, v47, v87, s55\nv_perm_b32 v63, v47, v87, s56\ns_bitcmp1_b32 s66, 0\n" \
  "s_cbranch_scc1 LRAWp_%=\nLRETp_%=:\ns_waitcnt lgkmcnt(0)\n.p2align 6\nLOOP_%=:\ns_waitcnt lgkmcnt(4)\n" \
  "v_mfma_f32_16x16x32_bf16 v[128:131], v[64:67], v[48:51], v[128:131]\nds_read_b64_tr_b16 v[76:77], v35 offset:0\n" \
  "ds_read_b64_tr_b16 v[78:79], v35 offset:1024\nv_mfma_f32_16x16x32_bf16 v[160:163], v[64:67], v[52:55], v[160:163]\ns_bitcmp1_b32 s64, 2\n" \
  "s_cselect_b32 s60, 0x80, 0\nv_and_or_b32 v23, v17, s60, v16\nbuffer_load_dwordx3 v[112:114], v15, s[40:43], s58 offen\n" \
  "v_mfma_f32_16x16x32_bf16 v[192:195], v[64:67], v[56:59], v[192:195]\nbuffer_load_ushort v115, v23, s[40:43], s58 offen\n" \
  "v_mfma_f32_16x16x32_bf16 v[224:227], v[64:67], v[60:63], v[224:227]\ns_waitcnt vmcnt(6)\nv_bfi_b32 v22, s57, v103, v20\nv_lshrrev_b32 v31, 9, v103\n" \
  "s_waitcnt lgkmcnt(4)\nv_mfma_f32_16x16x32_bf16 v[132:135], v[68:71], v[48:51], v[132:135]\nds_read_b64_tr_b16 v[64:65], v36 offset:0\n" \
  "ds_read_b64_tr_b16 v[66:67], v36 offset:1024\nv_mfma_f32_16x16x32_bf16 v[164:167], v[68:71], v[52:55], v[164:167]\nds_write_b8 v22, v31\n" \
  "ds_read_b128 v[40:43], v20\nv_mfma_f32_16x16x32_bf16 v[196:199], v[68:71], v[56:59], v[196:199]\n" \
  "buffer_load_dwordx4 v[104:107], v14, s[40:43], s58 offen\nds_read_b128 v[44:47], v20 offset:1024\nds_write_b8 v22, v21\n" \
  "v_mfma_f32_16x16x32_bf16 v[228:231], v[68:71], v[60:63], v[228:231]\nbuffer_load_dwordx4 v[108:111], v14, s[40:43], s58 offen offset:1024\n" \
  "s_add_u32 s58, s58, 47104\ns_waitcnt lgkmcnt(8)\nv_mfma_f32_16x16x32_bf16 v[136:139], v[72:75], v[48:51], v[136:139]\n" \
  "ds_read_b64_tr_b16 v[68:69], v37 offset:0\nds_read_b64_tr_b16 v[70:71], v37 offset:1024\n" \
  "v_mfma_f32_16x16x32_bf16 v[168:171], v[72:75], v[52:55], v[168:171]\nv_alignbit_b32 v25, v100, v100, 3\n" \
  "v_mfma_f32_16x16x32_bf16 v[200:203], v[72:75], v[56:59], v[200:203]\nv_alignbit_b32 v26, v101, v101, 3\nv_alignbit_b32 v27, v102, v102, 3\n" \
  "v_mfma_f32_16x16x32_bf16 v[232:235], v[72:75], v[60:63], v[232:235]\nv_lshrrev_b32 v28, 5, v100\nv_lshrrev_b32 v29, 5, v101\ns_waitcnt lgkmcnt(8)\n" \
  "v_mfma_f32_16x16x32_bf16 v[140:143], v[76:79], v[48:51], v[140:143]\nds_read_b64_tr_b16 v[72:73], v38 offset:0\n" \
  "ds_read_b64_tr_b16 v[74:75], v38 offset:1024\nv_mfma_f32_16x16x32_bf16 v[172:175], v[76:79], v[52:55], v[172:175]\nv_lshlrev_b32 v30, 2, v102\n" \
  "v_lshlrev_b32 v31, 1, v102\nv_mfma_f32_16x16x32_bf16 v[204:207], v[76:79], v[56:59], v[204:207]\ns_waitcnt lgkmcnt(5)\nv_bfi_b32 v40, s52, v100, v40\n" \
  "v_bfi_b32 v42, s52, v101, v42\nv_mfma_f32_16x16x32_bf16 v[236:239], v[76:79], v[60:63], v[236:239]\ns_add_u32 m0, s62, 32768\n" \
  "v_bfi_b32 v44, s52, v102, v44\nv_mfma_f32_16x16x32_bf16 v[144:147], v[64:67], v[48:51], v[144:147]\nds_read_b64_tr_b16 v[76:77], v39 offset:0\n" \
  "ds_read_b64_tr_b16 v[78:79], v39 offset:1024\nv_mfma_f32_16x16x32_bf16 v[176:179], v[64:67], v[52:55], v[176:179]\n" \
  "buffer_load_dwordx4 v18, s[44:47], s59 offen offset:0 lds\nv_bfi_b32 v41, s52, v25, v41\nv_bfi_b32 v43, s52, v26, v43\n" \
  "v_mfma_f32_16x16x32_bf16 v[208:211], v[64:67], v[56:59], v[208:211]\nbuffer_load_dwordx4 v19, s[44:47], s59 offen offset:1024 lds\n" \
  "s_add_u32 s59, s59, 262144\nv_bfi_b32 v45, s52, v27, v45\nv_bfi_b32 v46, s53, v28, v46\n" \
  "v_mfma_f32_16x16x32_bf16 v[240:243], v[64:67], v[60:63], v[240:243]\ns_waitcnt lgkmcnt(4)\n" \
  "v_mfma_f32_16x16x32_bf16 v[148:151], v[68:71], v[48:51], v[148:151]\nds_read_b64_tr_b16 v[64:65], v32 offset:8192\n" \
  "ds_read_b64_tr_b16 v[66:67], v32 offset:9216\nv_mfma_f32_16x16x32_bf16 v[180:183], v[68:71], v[52:55], v[180:183]\nv_bfi_b32 v47, s53, v29, v47\n" \
  "v_bfi_b32 v46, s54, v30, v46\nv_mfma_f32_16x16x32_bf16 v[212:215], v[68:71], v[56:59], v[212:215]\nv_bfi_b32 v47, s54, v31, v47\n" \
  "v_mfma_f32_16x16x32_bf16 v[244:247], v[68:71], v[60:63], v[244:247]\ns_waitcnt lgkmcnt(4)\n" \
  "v_mfma_f32_16x16x32_bf16 v[152:155], v[72:75], v[48:51], v[152:155]\nds_read_b64_tr_b16 v[68:69], v33 offset:8192\n" \
  "ds_read_b64_tr_b16 v[70:71], v33 offset:9216\nv_mfma_f32_16x16x32_bf16 v[184:187], v[72:75], v[52:55], v[184:187]\n" \
  "v_mfma_f32_16x16x32_bf16 v[216:219], v[72:75], v[56:59], v[216:219]\nv_mfma_f32_16x16x32_bf16 v[248:251], v[72:75], v[60:63], v[248:251]\n" \
  "s_waitcnt vmcnt(8)\ns_waitcnt lgkmcnt(4)\nv_mfma_f32_16x16x32_bf16 v[156:159], v[76:79], v[48:51], v[156:159]\n" \
  "ds_read_b64_tr_b16 v[72:73], v34 offset:8192\nds_read_b64_tr_b16 v[74:75], v34 offset:9216\ns_setprio 2\nv_perm_b32 v48, v40, v92, s55\n" \
  "v_perm_b32 v49, v40, v92, s56\nv_perm_b32 v50, v41, v93, s55\nv_perm_b32 v51, v41, v93, s56\n" \
  "v_mfma_f32_16x16x32_bf16 v[188:191], v[76:79], v[52:55], v[188:191]\nv_perm_b32 v52, v42, v94, s55\nv_perm_b32 v53, v42, v94, s56\n" \
  "v_perm_b32 v54, v43, v95, s55\nv_perm_b32 v55, v43, v95, s56\nv_mfma_f32_16x16x32_bf16 v[220:223], v[76:79], v[56:59], v[220:223]\n" \
  "v_perm_b32 v56, v44, v96, s55\nv_perm_b32 v57, v44, v96, s56\nv_perm_b32 v58, v45, v97, s55\nv_perm_b32 v59, v45, v97, s56\n" \
  "v_mfma_f32_16x16x32_bf16 v[252:255], v[76:79], v[60:63], v[252:255]\nv_perm_b32 v60, v46, v98, s55\nv_perm_b32 v61, v46, v98, s56\n" \
  "v_perm_b32 v62, v47, v99, s55\nv_perm_b32 v63, v47, v99, s56\ns_bitcmp1_b32 s66, 1\ns_cbranch_scc1 LRAW0_%=\nLRET0_%=:\ns_waitcnt lgkmcnt(4)\n" \
  "v_mfma_f32_16x16x32_bf16 v[128:131], v[64:67], v[48:51], v[128:131]\nds_read_b64_tr_b16 v[76:77], v35 offset:8192\n" \
  "ds_read_b64_tr_b16 v[78:79], v35 offset:9216\nv_mfma_f32_16x16x32_bf16 v[160:163], v[64:67], v[52:55], v[160:163]\ns_bitcmp1_b32 s64, 3\n" \
  "s_cselect_b32 s60, 0x80, 0\nv_and_or_b32 v23, v17, s60, v16\nbuffer_load_dwordx3 v[124:126], v15, s[40:43], s58 offen\n" \
  "v_mfma_f32_16x16x32_bf16 v[192:195], v[64:67], v[56:59], v[192:195]\nbuffer_load_ushort v127, v23, s[40:43], s58 offen\n" \
  "v_mfma_f32_16x16x32_bf16 v[224:227], v[64:67], v[60:63], v[224:227]\ns_waitcnt vmcnt(6)\nv_bfi_b32 v22, s57, v115, v20\nv_lshrrev_b32 v31, 9, v115\n" \
  "s_waitcnt lgkmcnt(4)\nv_mfma_f32_16x16x32_bf16 v[132:135], v[68:71], v[48:51], v[132:135]\nds_read_b64_tr_b16 v[64:65], v36 offset:8192\n" \
  "ds_read_b64_tr_b16 v[66:67], v36 offset:9216\nv_mfma_f32_16x16x32_bf16 v[164:167], v[68:71], v[52:55], v[164:167]\nds_write_b8 v22, v31\n" \
  "ds_read_b128 v[40:43], v20\nv_mfma_f32_16x16x32_bf16 v[196:199], v[68:71], v[56:59], v[196:199]\n" \
  "buffer_load_dwordx4 v[116:119], v14, s[40:43], s58 offen\nds_read_b128 v[44:47], v20 offset:1024\nds_write_b8 v22, v21\n" \
  "v_mfma_f32_16x16x32_bf16 v[228:231], v[68:71], v[60:63], v[228:231]\nbuffer_load_dwordx4 v[120:123], v14, s[40:43], s58 offen offset:1024\n" \
  "s_add_u32 s58, s58, 47104\ns_waitcnt lgkmcnt(8)\nv_mfma_f32_16x16x32_bf16 v[136:139], v[72:75], v[48:51], v[136:139]\n" \
  "ds_read_b64_tr_b16 v[68:69], v37 offset:8192\nds_read_b64_tr_b16 v[70:71], v37 offset:9216\ns_setprio 1\n" \
  "v_mfma_f32_16x16x32_bf16 v[168:171], v[72:75], v[52:55], v[168:171]\nv_alignbit_b32 v25, v112, v112, 3\n" \
  "v_mfma_f32_16x16x32_bf16 v[200:203], v[72:75], v[56:59], v[200:203]\nv_alignbit_b32 v26, v113, v113, 3\nv_alignbit_b32 v27, v114, v114, 3\n" \
  "v_mfma_f32_16x16x32_bf16 v[232:235], v[72:75], v[60:63], v[232:235]\nv_lshrrev_b32 v28, 5, v112\nv_lshrrev_b32 v29, 5, v113\ns_waitcnt lgkmcnt(8)\n" \
  "v_mfma_f32_16x16x32_bf16 v[140:143], v[76:79], v[48:51], v[140:143]\nds_read_b64_tr_b16 v[72:73], v38 offset:8192\n" \
  "ds_read_b64_tr_b16 v[74:75], v38 offset:9216\nv_mfma_f32_16x16x32_bf16 v[172:175], v[76:79], v[52:55], v[172:175]\nv_lshlrev_b32 v30, 2, v114\n" \
  "v_lshlrev_b32 v31, 1, v114\nv_mfma_f32_16x16x32_bf16 v[204:207], v[76:79], v[56:59], v[204:207]\ns_waitcnt lgkmcnt(5)\nv_bfi_b32 v40, s52, v112, v40\n" \
  "v_bfi_b32 v42, s52, v113, v42\nv_mfma_f32_16x16x32_bf16 v[236:239], v[76:79], v[60:63], v[236:239]\ns_add_u32 m0, s62, 40960\n" \
  "v_bfi_b32 v44, s52, v114, v44\nv_mfma_f32_16x16x32_bf16 v[144:147], v[64:67], v[48:51], v[144:147]\nds_read_b64_tr_b16 v[76:77], v39 offset:8192\n" \
  "ds_read_b64_tr_b16 v[78:79], v39 offset:9216\ns_setprio 0\nv_mfma_f32_16x16x32_bf16 v[176:179], v[64:67], v[52:55], v[176:179]\n" \
  "buffer_load_dwordx4 v18, s[44:47], s59 offen offset:0 lds\nv_bfi_b32 v41, s52, v25, v41\nv_bfi_b32 v43, s52, v26, v43\n" \
  "v_mfma_f32_16x16x32_bf16 v[208:211], v[64:67], v[56:59], v[208:211]\nbuffer_load_dwordx4 v19, s[44:47], s59 offen offset:1024 lds\n" \
  "s_add_u32 s59, s59, 262144\nv_bfi_b32 v45, s52, v27, v45\nv_bfi_b32 v46, s53, v28, v46\n" \
  "v_mfma_f32_16x16x32_bf16 v[240:243], v[64:67], v[60:63], v[240:243]\ns_waitcnt lgkmcnt(4)\n" \
  "v_mfma_f32_16x16x32_bf16 v[148:151], v[68:71], v[48:51], v[148:151]\ns_barrier\nds_read_b64_tr_b16 v[64:65], v32 offset:16384\n" \
  "ds_read_b64_tr_b16 v[66:67], v32 offset:17408\ns_setprio 3\nv_mfma_f32_16x16x32_bf16 v[180:183], v[68:71], v[52:55], v[180:183]\n" \
  "v_bfi_b32 v47, s53, v29, v47\nv_bfi_b32 v46, s54, v30, v46\nv_mfma_f32_16x16x32_bf16 v[212:215], v[68:71], v[56:59], v[212:215]\n" \
  "v_bfi_b32 v47, s54, v31, v47\nv_mfma_f32_16x16x32_bf16 v[244:247], v[68:71], v[60:63], v[244:247]\ns_waitcnt lgkmcnt(4)\n" \
  "v_mfma_f32_16x16x32_bf16 v[152:155], v[72:75], v[48:51], v[152:155]\nds_read_b64_tr_b16 v[68:69], v33 offset:16384\n" \
  "ds_read_b64_tr_b16 v[70:71], v33 offset:17408\nv_mfma_f32_16x16x32_bf16 v[184:187], v[72:75], v[52:55], v[184:187]\n" \
  "v_mfma_f32_16x16x32_bf16 v[216:219], v[72:75], v[56:59], v[216:219]\nv_mfma_f32_16x16x32_bf16 v[248:251], v[72:75], v[60:63], v[248:251]\n" \
  "s_waitcnt vmcnt(8)\ns_waitcnt lgkmcnt(4)\nv_mfma_f32_16x16x32_bf16 v[156:159], v[76:79], v[48:51], v[156:159]\n" \
  "ds_read_b64_tr_b16 v[72:73], v34 offset:16384\nds_read_b64_tr_b16 v[74:75], v34 offset:17408\nv_perm_b32 v48, v40, v104, s55\n" \
  "v_perm_b32 v49, v40, v104, s56\nv_perm_b32 v50, v41, v105, s55\nv_perm_b32 v51, v41, v105, s56\n" \
  "v_mfma_f32_16x16x32_bf16 v[188:191], v[76:79], v[52:55], v[188:191]\nv_perm_b32 v52, v42, v106, s55\nv_perm_b32 v53, v42, v106, s56\n" \
  "v_perm_b32 v54, v43, v107, s55\nv_perm_b32 v55, v43, v107, s56\nv_mfma_f32_16x16x32_bf16 v[220:223], v[76:79], v[56:59], v[220:223]\n" \
  "v_perm_b32 v56, v44, v108, s55\nv_perm_b32 v57, v44, v108, s56\nv_perm_b32 v58, v45, v109, s55\nv_perm_b32 v59, v45, v109, s56\n" \
  "v_mfma_f32_16x16x32_bf16 v[252:255], v[76:79], v[60:63], v[252:255]\nv_perm_b32 v60, v46, v110, s55\nv_perm_b32 v61, v46, v110, s56\n" \
  "v_perm_b32 v62, v47, v111, s55\nv_perm_b32 v63, v47, v111, s56\ns_bitcmp1_b32 s66, 2\ns_cbranch_scc1 LRAW1_%=\nLRET1_%=:\ns_waitcnt lgkmcnt(4)\n" \
  "v_mfma_f32_16x16x32_bf16 v[128:131], v[64:67], v[48:51], v[128:131]\nds_read_b64_tr_b16 v[76:77], v35 offset:16384\n" \
  "ds_read_b64_tr_b16 v[78:79], v35 offset:17408\nv_mfma_f32_16x16x32_bf16 v[160:163], v[64:67], v[52:55], v[160:163]\ns_bitcmp1_b32 s64, 4\n" \
  "s_cselect_b32 s60, 0x80, 0\nv_and_or_b32 v23, v17, s60, v16\nbuffer_load_dwordx3 v[88:90], v15, s[40:43], s58 offen\n" \
  "v_mfma_f32_16x16x32_bf16 v[192:195], v[64:67], v[56:59], v[192:195]\nbuffer_load_ushort v91, v23, s[40:43], s58 offen\n" \
  "v_mfma_f32_16x16x32_bf16 v[224:227], v[64:67], v[60:63], v[224:227]\ns_waitcnt vmcnt(6)\nv_bfi_b32 v22, s57, v127, v20\nv_lshrrev_b32 v31, 9, v127\n" \
  "s_waitcnt lgkmcnt(4)\nv_mfma_f32_16x16x32_bf16 v[132:135], v[68:71], v[48:51], v[132:135]\nds_read_b64_tr_b16 v[64:65], v36 offset:16384\n" \
  "ds_read_b64_tr_b16 v[66:67], v36 offset:17408\nv_mfma_f32_16x16x32_bf16 v[164:167], v[68:71], v[52:55], v[164:167]\nds_write_b8 v22, v31\n" \
  "ds_read_b128 v[40:43], v20\nv_mfma_f32_16x16x32_bf16 v[196:199], v[68:71], v[56:59], v[196:199]\n" \
  "buffer_load_dwordx4 v[80:83], v14, s[40:43], s58 offen\nds_read_b128 v[44:47], v20 offset:1024\nds_write_b8 v22, v21\n" \
  "v_mfma_f32_16x16x32_bf16 v[228:231], v[68:71], v[60:63], v[228:231]\nbuffer_load_dwordx4 v[84:87], v14, s[40:43], s58 offen offset:1024\n" \
  "s_add_u32 s58, s58, 47104\ns_waitcnt lgkmcnt(8)\nv_mfma_f32_16x16x32_bf16 v[136:139], v[72:75], v[48:51], v[136:139]\n" \
  "ds_read_b64_tr_b16 v[68:69], v37 offset:16384\nds_read_b64_tr_b16 v[70:71], v37 offset:17408\n" \
  "v_mfma_f32_16x16x32_bf16 v[168:171], v[72:75], v[52:55], v[168:171]\nv_alignbit_b32 v25, v124, v124, 3\n" \
  "v_mfma_f32_16x16x32_bf16 v[200:203], v[72:75], v[56:59], v[200:203]\nv_alignbit_b32 v26, v125, v125, 3\nv_alignbit_b32 v27, v126, v126, 3\n" \
  "v_mfma_f32_16x16x32_bf16 v[232:235], v[72:75], v[60:63], v[232:235]\nv_lshrrev_b32 v28, 5, v124\nv_lshrrev_b32 v29, 5, v125\ns_waitcnt lgkmcnt(8)\n" \
  "v_mfma_f32_16x16x32_bf16 v[140:143], v[76:79], v[48:51], v[140:143]\nds_read_b64_tr_b16 v[72:73], v38 offset:16384\n" \
  "ds_read_b64_tr_b16 v[74:75], v38 offset:17408\nv_mfma_f32_16x16x32_bf16 v[172:175], v[76:79], v[52:55], v[172:175]\nv_lshlrev_b32 v30, 2, v126\n" \
  "v_lshlrev_b32 v31, 1, v126\nv_mfma_f32_16x16x32_bf16 v[204:207], v[76:79], v[56:59], v[204:207]\ns_waitcnt lgkmcnt(5)\nv_bfi_b32 v40, s52, v124, v40\n" \
  "v_bfi_b32 v42, s52, v125, v42\nv_mfma_f32_16x16x32_bf16 v[236:239], v[76:79], v[60:63], v[236:239]\ns_add_u32 m0, s62, 49152\n" \
  "v_bfi_b32 v44, s52, v126, v44\nv_mfma_f32_16x16x32_bf16 v[144:147], v[64:67], v[48:51], v[144:147]\nds_read_b64_tr_b16 v[76:77], v39 offset:16384\n" \
  "ds_read_b64_tr_b16 v[78:79], v39 offset:17408\nv_mfma_f32_16x16x32_bf16 v[176:179], v[64:67], v[52:55], v[176:179]\n" \
  "buffer_load_dwordx4 v18, s[44:47], s59 offen offset:0 lds\nv_bfi_b32 v41, s52, v25, v41\nv_bfi_b32 v43, s52, v26, v43\n" \
  "v_mfma_f32_16x16x32_bf16 v[208:211], v[64:67], v[56:59], v[208:211]\nbuffer_load_dwordx4 v19, s[44:47], s59 offen offset:1024 lds\n" \
  "s_add_u32 s59, s59, 262144\nv_bfi_b32 v45, s52, v27, v45\nv_bfi_b32 v46, s53, v28, v46\n" \
  "v_mfma_f32_16x16x32_bf16 v[240:243], v[64:67], v[60:63], v[240:243]\ns_waitcnt lgkmcnt(4)\n" \
  "v_mfma_f32_16x16x32_bf16 v[148:151], v[68:71], v[48:51], v[148:151]\nds_read_b64_tr_b16 v[64:65], v32 offset:24576\n" \
  "ds_read_b64_tr_b16 v[66:67], v32 offset:25600\nv_mfma_f32_16x16x32_bf16 v[180:183], v[68:71], v[52:55], v[180:183]\nv_bfi_b32 v47, s53, v29, v47\n" \
  "v_bfi_b32 v46, s54, v30, v46\nv_mfma_f32_16x16x32_bf16 v[212:215], v[68:71], v[56:59], v[212:215]\nv_bfi_b32 v47, s54, v31, v47\n" \
  "v_mfma_f32_16x16x32_bf16 v[244:247], v[68:71], v[60:63], v[244:247]\ns_waitcnt lgkmcnt(4)\n" \
  "v_mfma_f32_16x16x32_bf16 v[152:155], v[72:75], v[48:51], v[152:155]\nds_read_b64_tr_b16 v[68:69], v33 offset:24576\n" \
  "ds_read_b64_tr_b16 v[70:71], v33 offset:25600\nv_mfma_f32_16x16x32_bf16 v[184:187], v[72:75], v[52:55], v[184:187]\n" \
  "v_mfma_f32_16x16x32_bf16 v[216:219], v[72:75], v[56:59], v[216:219]\nv_mfma_f32_16x16x32_bf16 v[248:251], v[72:75], v[60:63], v[248:251]\n" \
  "s_waitcnt vmcnt(8)\ns_waitcnt lgkmcnt(4)\nv_mfma_f32_16x16x32_bf16 v[156:159], v[76:79], v[48:51], v[156:159]\n" \
  "ds_read_b64_tr_b16 v[72:73], v34 offset:24576\nds_read_b64_tr_b16 v[74:75], v34 offset:25600\ns_setprio 2\nv_perm_b32 v48, v40, v116, s55\n" \
  "v_perm_b32 v49, v40, v116, s56\nv_perm_b32 v50, v41, v117, s55\nv_perm_b32 v51, v41, v117, s56\n" \
  "v_mfma_f32_16x16x32_bf16 v[188:191], v[76:79], v[52:55], v[188:191]\nv_perm_b32 v52, v42, v118, s55\nv_perm_b32 v53, v42, v118, s56\n" \
  "v_perm_b32 v54, v43, v119, s55\nv_perm_b32 v55, v43, v119, s56\nv_mfma_f32_16x16x32_bf16 v[220:223], v[76:79], v[56:59], v[220:223]\n" \
  "v_perm_b32 v56, v44, v120, s55\nv_perm_b32 v57, v44, v120, s56\nv_perm_b32 v58, v45, v121, s55\nv_perm_b32 v59, v45, v121, s56\n" \
  "v_mfma_f32_16x16x32_bf16 v[252:255], v[76:79], v[60:63], v[252:255]\nv_perm_b32 v60, v46, v122, s55\nv_perm_b32 v61, v46, v122, s56\n" \
  "v_perm_b32 v62, v47, v123, s55\nv_perm_b32 v63, v47, v123, s56\ns_bitcmp1_b32 s66, 3\ns_cbranch_scc1 LRAW2_%=\nLRET2_%=:\ns_waitcnt lgkmcnt(4)\n" \
  "v_mfma_f32_16x16x32_bf16 v[128:131], v[64:67], v[48:51], v[128:131]\nds_read_b64_tr_b16 v[76:77], v35 offset:24576\n" \
  "ds_read_b64_tr_b16 v[78:79], v35 offset:25600\nv_mfma_f32_16x16x32_bf16 v[160:163], v[64:67], v[52:55], v[160:163]\ns_bitcmp1_b32 s64, 5\n" \
  "s_cselect_b32 s60, 0x80, 0\nv_and_or_b32 v23, v17, s60, v16\nbuffer_load_dwordx3 v[100:102], v15, s[40:43], s58 offen\n" \
  "v_mfma_f32_16x16x32_bf16 v[192:195], v[64:67], v[56:59], v[192:195]\nbuffer_load_ushort v103, v23, s[40:43], s58 offen\n" \
  "v_mfma_f32_16x16x32_bf16 v[224:227], v[64:67], v[60:63], v[224:227]\ns_waitcnt vmcnt(6)\nv_bfi_b32 v22, s57, v91, v20\nv_lshrrev_b32 v31, 9, v91\n" \
  "s_waitcnt lgkmcnt(4)\nv_mfma_f32_16x16x32_bf16 v[132:135], v[68:71], v[48:51], v[132:135]\nds_read_b64_tr_b16 v[64:65], v36 offset:24576\n" \
  "ds_read_b64_tr_b16 v[66:67], v36 offset:25600\nv_mfma_f32_16x16x32_bf16 v[164:167], v[68:71], v[52:55], v[164:167]\nds_write_b8 v22, v31\n" \
  "ds_read_b128 v[40:43], v20\nv_mfma_f32_16x16x32_bf16 v[196:199], v[68:71], v[56:59], v[196:199]\n" \
  "buffer_load_dwordx4 v[92:95], v14, s[40:43], s58 offen\nds_read_b128 v[44:47], v20 offset:1024\nds_write_b8 v22, v21\n" \
  "v_mfma_f32_16x16x32_bf16 v[228:231], v[68:71], v[60:63], v[228:231]\nbuffer_load_dwordx4 v[96:99], v14, s[40:43], s58 offen offset:1024\n" \
  "s_add_u32 s58, s58, 47104\ns_waitcnt lgkmcnt(8)\nv_mfma_f32_16x16x32_bf16 v[136:139], v[72:75], v[48:51], v[136:139]\n" \
  "ds_read_b64_tr_b16 v[68:69], v37 offset:24576\nds_read_b64_tr_b16 v[70:71], v37 offset:25600\ns_setprio 1\n" \
  "v_mfma_f32_16x16x32_bf16 v[168:171], v[72:75], v[52:55], v[168:171]\nv_alignbit_b32 v25, v88, v88, 3\n" \
  "v_mfma_f32_16x16x32_bf16 v[200:203], v[72:75], v[56:59], v[200:203]\nv_alignbit_b32 v26, v89, v89, 3\nv_alignbit_b32 v27, v90, v90, 3\n" \
  "v_mfma_f32_16x16x32_bf16 v[232:235], v[72:75], v[60:63], v[232:235]\nv_lshrrev_b32 v28, 5, v88\nv_lshrrev_b32 v29, 5, v89\ns_waitcnt lgkmcnt(8)\n" \
  "v_mfma_f32_16x16x32_bf16 v[140:143], v[76:79], v[48:51], v[140:143]\nds_read_b64_tr_b16 v[72:73], v38 offset:24576\n" \
  "ds_read_b64_tr_b16 v[74:75], v38 offset:25600\nv_mfma_f32_16x16x32_bf16 v[172:175], v[76:79], v[52:55], v[172:175]\nv_lshlrev_b32 v30, 2, v90\n" \
  "v_lshlrev_b32 v31, 1, v90\nv_mfma_f32_16x16x32_bf16 v[204:207], v[76:79], v[56:59], v[204:207]\ns_waitcnt lgkmcnt(5)\nv_bfi_b32 v40, s52, v88, v40\n" \
  "v_bfi_b32 v42, s52, v89, v42\nv_mfma_f32_16x16x32_bf16 v[236:239], v[76:79], v[60:63], v[236:239]\ns_add_u32 m0, s62, 57344\n" \
  "v_bfi_b32 v44, s52, v90, v44\nv_mfma_f32_16x16x32_bf16 v[144:147], v[64:67], v[48:51], v[144:147]\nds_read_b64_tr_b16 v[76:77], v39 offset:24576\n" \
  "ds_read_b64_tr_b16 v[78:79], v39 offset:25600\ns_setprio 0\nv_mfma_f32_16x16x32_bf16 v[176:179], v[64:67], v[52:55], v[176:179]\n" \
  "buffer_load_dwordx4 v18, s[44:47], s59 offen offset:0 lds\nv_bfi_b32 v41, s52, v25, v41\nv_bfi_b32 v43, s52, v26, v43\n" \
  "v_mfma_f32_16x16x32_bf16 v[208:211], v[64:67], v[56:59], v[208:211]\nbuffer_load_dwordx4 v19, s[44:47], s59 offen offset:1024 lds\n" \
  "s_add_u32 s59, s59, 262144\nv_bfi_b32 v45, s52, v27, v45\nv_bfi_b32 v46, s53, v28, v46\n" \
  "v_mfma_f32_16x16x32_bf16 v[240:243], v[64:67], v[60:63], v[240:243]\ns_waitcnt lgkmcnt(4)\n" \
  "v_mfma_f32_16x16x32_bf16 v[148:151], v[68:71], v[48:51], v[148:151]\ns_barrier\nds_read_b64_tr_b16 v[64:65], v32 offset:32768\n" \
  "ds_read_b64_tr_b16 v[66:67], v32 offset:33792\ns_setprio 3\nv_mfma_f32_16x16x32_bf16 v[180:183], v[68:71], v[52:55], v[180:183]\n" \
  "v_bfi_b32 v47, s53, v29, v47\nv_bfi_b32 v46, s54, v30, v46\nv_mfma_f32_16x16x32_bf16 v[212:215], v[68:71], v[56:59], v[212:215]\n" \
  "v_bfi_b32 v47, s54, v31, v47\nv_mfma_f32_16x16x32_bf16 v[244:247], v[68:71], v[60:63], v[244:247]\ns_waitcnt lgkmcnt(4)\n" \
  "v_mfma_f32_16x16x32_bf16 v[152:155], v[72:75], v[48:51], v[152:155]\nds_read_b64_tr_b16 v[68:69], v33 offset:32768\n" \
  "ds_read_b64_tr_b16 v[70:71], v33 offset:33792\nv_mfma_f32_16x16x32_bf16 v[184:187], v[72:75], v[52:55], v[184:187]\n" \
  "v_mfma_f32_16x16x32_bf16 v[216:219], v[72:75], v[56:59], v[216:219]\nv_mfma_f32_16x16x32_bf16 v[248:251], v[72:75], v[60:63], v[248:251]\n" \
  "s_waitcnt vmcnt(8)\ns_waitcnt lgkmcnt(4)\nv_mfma_f32_16x16x32_bf16 v[156:159], v[76:79], v[48:51], v[156:159]\n" \
  "ds_read_b64_tr_b16 v[72:73], v34 offset:32768\nds_read_b64_tr_b16 v[74:75], v34 offset:33792\nv_perm_b32 v48, v40, v80, s55\n" \
  "v_perm_b32 v49, v40, v80, s56\nv_perm_b32 v50, v41, v81, s55\nv_perm_b32 v51, v41, v81, s56\n" \
  "v_mfma_f32_16x16x32_bf16 v[188:191], v[76:79], v[52:55], v[188:191]\nv_perm_b32 v52, v42, v82, s55\nv_perm_b32 v53, v42, v82, s56\n" \
  "v_perm_b32 v54, v43, v83, s55\nv_perm_b32 v55, v43, v83, s56\nv_mfma_f32_16x16x32_bf16 v[220:223], v[76:79], v[56:59], v[220:223]\n" \
  "v_perm_b32 v56, v44, v84, s55\nv_perm_b32 v57, v44, v84, s56\nv_perm_b32 v58, v45, v85, s55\nv_perm_b32 v59, v45, v85, s56\n" \
  "v_mfma_f32_16x16x32_bf16 v[252:255], v[76:79], v[60:63], v[252:255]\nv_perm_b32 v60, v46, v86, s55\nv_perm_b32 v61, v46, v86, s56\n" \
  "v_perm_b32 v62, v47, v87, s55\nv_perm_b32 v63, v47, v87, s56\ns_bitcmp1_b32 s66, 4\ns_cbranch_scc1 LRAW3_%=\nLRET3_%=:\ns_waitcnt lgkmcnt(4)\n" \
  "v_mfma_f32_16x16x32_bf16 v[128:131], v[64:67], v[48:51], v[128:131]\nds_read_b64_tr_b16 v[76:77], v35 offset:32768\n" \
  "ds_read_b64_tr_b16 v[78:79], v35 offset:33792\nv_mfma_f32_16x16x32_bf16 v[160:163], v[64:67], v[52:55], v[160:163]\ns_bitcmp1_b32 s64, 6\n" \
  "s_cselect_b32 s60, 0x80, 0\nv_and_or_b32 v23, v17, s60, v16\nbuffer_load_dwordx3 v[112:114], v15, s[40:43], s58 offen\n" \
  "v_mfma_f32_16x16x32_bf16 v[192:195], v[64:67], v[56:59], v[192:195]\nbuffer_load_ushort v115, v23, s[40:43], s58 offen\ns_cmp_eq_u32 s61, 7\n" \
  "s_cselect_b32 s46, 0, s46\nv_mfma_f32_16x16x32_bf16 v[224:227], v[64:67], v[60:63], v[224:227]\ns_waitcnt vmcnt(6)\nv_bfi_b32 v22, s57, v103, v20\n" \
  "v_lshrrev_b32 v31, 9, v103\ns_waitcnt lgkmcnt(4)\nv_mfma_f32_16x16x32_bf16 v[132:135], v[68:71], v[48:51], v[132:135]\n" \
  "ds_read_b64_tr_b16 v[64:65], v36 offset:32768\nds_read_b64_tr_b16 v[66:67], v36 offset:33792\n" \
  "v_mfma_f32_16x16x32_bf16 v[164:167], v[68:71], v[52:55], v[164:167]\nds_write_b8 v22, v31\nds_read_b128 v[40:43], v20\n" \
  "v_mfma_f32_16x16x32_bf16 v[196:199], v[68:71], v[56:59], v[196:199]\nbuffer_load_dwordx4 v[104:107], v14, s[40:43], s58 offen\n" \
  "ds_read_b128 v[44:47], v20 offset:1024\nds_write_b8 v22, v21\nv_mfma_f32_16x16x32_bf16 v[228:231], v[68:71], v[60:63], v[228:231]\n" \
  "buffer_load_dwordx4 v[108:111], v14, s[40:43], s58 offen offset:1024\ns_add_u32 s58, s58, 47104\ns_waitcnt lgkmcnt(8)\n" \
  "v_mfma_f32_16x16x32_bf16 v[136:139], v[72:75], v[48:51], v[136:139]\nds_read_b64_tr_b16 v[68:69], v37 offset:32768\n" \
  "ds_read_b64_tr_b16 v[70:71], v37 offset:33792\nv_mfma_f32_16x16x32_bf16 v[168:171], v[72:75], v[52:55], v[168:171]\n" \
  "v_alignbit_b32 v25, v100, v100, 3\nv_mfma_f32_16x16x32_bf16 v[200:203], v[72:75], v[56:59], v[200:203]\nv_alignbit_b32 v26, v101, v101, 3\n" \
  "v_alignbit_b32 v27, v102, v102, 3\nv_mfma_f32_16x16x32_bf16 v[232:235], v[72:75], v[60:63], v[232:235]\nv_lshrrev_b32 v28, 5, v100\n" \
  "v_lshrrev_b32 v29, 5, v101\ns_waitcnt lgkmcnt(8)\nv_mfma_f32_16x16x32_bf16 v[140:143], v[76:79], v[48:51], v[140:143]\n" \
  "ds_read_b64_tr_b16 v[72:73], v38 offset:32768\nds_read_b64_tr_b16 v[74:75], v38 offset:33792\n" \
  "v_mfma_f32_16x16x32_bf16 v[172:175], v[76:79], v[52:55], v[172:175]\nv_lshlrev_b32 v30, 2, v102\nv_lshlrev_b32 v31, 1, v102\n" \
  "v_mfma_f32_16x16x32_bf16 v[204:207], v[76:79], v[56:59], v[204:207]\ns_waitcnt lgkmcnt(5)\nv_bfi_b32 v40, s52, v100, v40\n" \
  "v_bfi_b32 v42, s52, v101, v42\nv_mfma_f32_16x16x32_bf16 v[236:239], v[76:79], v[60:63], v[236:239]\ns_add_u32 m0, s62, 0\n" \
  "v_bfi_b32 v44, s52, v102, v44\nv_mfma_f32_16x16x32_bf16 v[144:147], v[64:67], v[48:51], v[144:147]\nds_read_b64_tr_b16 v[76:77], v39 offset:32768\n" \
  "ds_read_b64_tr_b16 v[78:79], v39 offset:33792\nv_mfma_f32_16x16x32_bf16 v[176:179], v[64:67], v[52:55], v[176:179]\n" \
  "buffer_load_dwordx4 v18, s[44:47], s59 offen offset:0 lds\nv_bfi_b32 v41, s52, v25, v41\nv_bfi_b32 v43, s52, v26, v43\n" \
  "v_mfma_f32_16x16x32_bf16 v[208:211], v[64:67], v[56:59], v[208:211]\nbuffer_load_dwordx4 v19, s[44:47], s59 offen offset:1024 lds\n" \
  "s_add_u32 s59, s59, 262144\nv_bfi_b32 v45, s52, v27, v45\nv_bfi_b32 v46, s53, v28, v46\n" \
  "v_mfma_f32_16x16x32_bf16 v[240:243], v[64:67], v[60:63], v[240:243]\ns_waitcnt lgkmcnt(4)\n" \
  "v_mfma_f32_16x16x32_bf16 v[148:151], v[68:71], v[48:51], v[148:151]\nds_read_b64_tr_b16 v[64:65], v32 offset:40960\n" \
  "ds_read_b64_tr_b16 v[66:67], v32 offset:41984\nv_mfma_f32_16x16x32_bf16 v[180:183], v[68:71], v[52:55], v[180:183]\nv_bfi_b32 v47, s53, v29, v47\n" \
  "v_bfi_b32 v46, s54, v30, v46\nv_mfma_f32_16x16x32_bf16 v[212:215], v[68:71], v[56:59], v[212:215]\nv_bfi_b32 v47, s54, v31, v47\n" \
  "v_mfma_f32_16x16x32_bf16 v[244:247], v[68:71], v[60:63], v[244:247]\ns_waitcnt lgkmcnt(4)\n" \
  "v_mfma_f32_16x16x32_bf16 v[152:155], v[72:75], v[48:51], v[152:155]\nds_read_b64_tr_b16 v[68:69], v33 offset:40960\n" \
  "ds_read_b64_tr_b16 v[70:71], v33 offset:41984\nv_mfma_f32_16x16x32_bf16 v[184:187], v[72:75], v[52:55], v[184:187]\n" \
  "v_mfma_f32_16x16x32_bf16 v[216:219], v[72:75], v[56:59], v[216:219]\nv_mfma_f32_16x16x32_bf16 v[248:251], v[72:75], v[60:63], v[248:251]\n" \
  "s_waitcnt vmcnt(8)\ns_waitcnt lgkmcnt(4)\nv_mfma_f32_16x16x32_bf16 v[156:159], v[76:79], v[48:51], v[156:159]\n" \
  "ds_read_b64_tr_b16 v[72:73], v34 offset:40960\nds_read_b64_tr_b16 v[74:75], v34 offset:41984\ns_setprio 2\nv_perm_b32 v48, v40, v92, s55\n" \
  "v_perm_b32 v49, v40, v92, s56\nv_perm_b32 v50, v41, v93, s55\nv_perm_b32 v51, v41, v93, s56\n" \
  "v_mfma_f32_16x16x32_bf16 v[188:191], v[76:79], v[52:55], v[188:191]\nv_perm_b32 v52, v42, v94, s55\nv_perm_b32 v53, v42, v94, s56\n" \
  "v_perm_b32 v54, v43, v95, s55\nv_perm_b32 v55, v43, v95, s56\nv_mfma_f32_16x16x32_bf16 v[220:223], v[76:79], v[56:59], v[220:223]\n" \
  "v_perm_b32 v56, v44, v96, s55\nv_perm_b32 v57, v44, v96, s56\nv_perm_b32 v58, v45, v97, s55\nv_perm_b32 v59, v45, v97, s56\n" \
  "v_mfma_f32_16x16x32_bf16 v[252:255], v[76:79], v[60:63], v[252:255]\nv_perm_b32 v60, v46, v98, s55\nv_perm_b32 v61, v46, v98, s56\n" \
  "v_perm_b32 v62, v47, v99, s55\nv_perm_b32 v63, v47, v99, s56\ns_bitcmp1_b32 s66, 5\ns_cbranch_scc1 LRAW4_%=\nLRET4_%=:\ns_waitcnt lgkmcnt(4)\n" \
  "v_mfma_f32_16x16x32_bf16 v[128:131], v[64:67], v[48:51], v[128:131]\nds_read_b64_tr_b16 v[76:77], v35 offset:40960\n" \
  "ds_read_b64_tr_b16 v[78:79], v35 offset:41984\nv_mfma_f32_16x16x32_bf16 v[160:163], v[64:67], v[52:55], v[160:163]\ns_bitcmp1_b32 s64, 7\n" \
  "s_cselect_b32 s60, 0x80, 0\nv_and_or_b32 v23, v17, s60, v16\nbuffer_load_dwordx3 v[124:126], v15, s[40:43], s58 offen\n" \
  "v_mfma_f32_16x16x32_bf16 v[192:195], v[64:67], v[56:59], v[192:195]\nbuffer_load_ushort v127, v23, s[40:43], s58 offen\n" \
  "v_mfma_f32_16x16x32_bf16 v[224:227], v[64:67], v[60:63], v[224:227]\ns_waitcnt vmcnt(6)\nv_bfi_b32 v22, s57, v115, v20\nv_lshrrev_b32 v31, 9, v115\n" \
  "s_waitcnt lgkmcnt(4)\nv_mfma_f32_16x16x32_bf16 v[132:135], v[68:71], v[48:51], v[132:135]\nds_read_b64_tr_b16 v[64:65], v36 offset:40960\n" \
  "ds_read_b64_tr_b16 v[66:67], v36 offset:41984\nv_mfma_f32_16x16x32_bf16 v[164:167], v[68:71], v[52:55], v[164:167]\nds_write_b8 v22, v31\n" \
  "ds_read_b128 v[40:43], v20\nv_mfma_f32_16x16x32_bf16 v[196:199], v[68:71], v[56:59], v[196:199]\n" \
  "buffer_load_dwordx4 v[116:119], v14, s[40:43], s58 offen\nds_read_b128 v[44:47], v20 offset:1024\nds_write_b8 v22, v21\n" \
  "v_mfma_f32_16x16x32_bf16 v[228:231], v[68:71], v[60:63], v[228:231]\nbuffer_load_dwordx4 v[120:123], v14, s[40:43], s58 offen offset:1024\n" \
  "s_add_u32 s58, s58, 47104\ns_cmp_eq_u32 s61, 7\ns_cselect_b32 s42, 0, s42\ns_waitcnt lgkmcnt(8)\n" \
  "v_mfma_f32_16x16x32_bf16 v[136:139], v[72:75], v[48:51], v[136:139]\nds_read_b64_tr_b16 v[68:69], v37 offset:40960\n" \
  "ds_read_b64_tr_b16 v[70:71], v37 offset:41984\ns_setprio 1\nv_mfma_f32_16x16x32_bf16 v[168:171], v[72:75], v[52:55], v[168:171]\n" \
  "v_alignbit_b32 v25, v112, v112, 3\nv_mfma_f32_16x16x32_bf16 v[200:203], v[72:75], v[56:59], v[200:203]\nv_alignbit_b32 v26, v113, v113, 3\n" \
  "v_alignbit_b32 v27, v114, v114, 3\nv_mfma_f32_16x16x32_bf16 v[232:235], v[72:75], v[60:63], v[232:235]\nv_lshrrev_b32 v28, 5, v112\n" \
  "v_lshrrev_b32 v29, 5, v113\ns_waitcnt lgkmcnt(8)\nv_mfma_f32_16x16x32_bf16 v[140:143], v[76:79], v[48:51], v[140:143]\n" \
  "ds_read_b64_tr_b16 v[72:73], v38 offset:40960\nds_read_b64_tr_b16 v[74:75], v38 offset:41984\n" \
  "v_mfma_f32_16x16x32_bf16 v[172:175], v[76:79], v[52:55], v[172:175]\nv_lshlrev_b32 v30, 2, v114\nv_lshlrev_b32 v31, 1, v114\n" \
  "v_mfma_f32_16x16x32_bf16 v[204:207], v[76:79], v[56:59], v[204:207]\ns_waitcnt lgkmcnt(5)\nv_bfi_b32 v40, s52, v112, v40\n" \
  "v_bfi_b32 v42, s52, v113, v42\nv_mfma_f32_16x16x32_bf16 v[236:239], v[76:79], v[60:63], v[236:239]\ns_add_u32 m0, s62, 8192\n" \
  "v_bfi_b32 v44, s52, v114, v44\nv_mfma_f32_16x16x32_bf16 v[144:147], v[64:67], v[48:51], v[144:147]\nds_read_b64_tr_b16 v[76:77], v39 offset:40960\n" \
  "ds_read_b64_tr_b16 v[78:79], v39 offset:41984\ns_setprio 0\nv_mfma_f32_16x16x32_bf16 v[176:179], v[64:67], v[52:55], v[176:179]\n" \
  "buffer_load_dwordx4 v18, s[44:47], s59 offen offset:0 lds\nv_bfi_b32 v41, s52, v25, v41\nv_bfi_b32 v43, s52, v26, v43\n" \
  "v_mfma_f32_16x16x32_bf16 v[208:211], v[64:67], v[56:59], v[208:211]\nbuffer_load_dwordx4 v19, s[44:47], s59 offen offset:1024 lds\n" \
  "s_add_u32 s59, s59, 262144\nv_bfi_b32 v45, s52, v27, v45\nv_bfi_b32 v46, s53, v28, v46\n" \
  "v_mfma_f32_16x16x32_bf16 v[240:243], v[64:67], v[60:63], v[240:243]\ns_waitcnt lgkmcnt(4)\n" \
  "v_mfma_f32_16x16x32_bf16 v[148:151], v[68:71], v[48:51], v[148:151]\ns_barrier\nds_read_b64_tr_b16 v[64:65], v32 offset:49152\n" \
  "ds_read_b64_tr_b16 v[66:67], v32 offset:50176\ns_setprio 3\nv_mfma_f32_16x16x32_bf16 v[180:183], v[68:71], v[52:55], v[180:183]\n" \
  "v_bfi_b32 v47, s53, v29, v47\nv_bfi_b32 v46, s54, v30, v46\nv_mfma_f32_16x16x32_bf16 v[212:215], v[68:71], v[56:59], v[212:215]\n" \
  "v_bfi_b32 v47, s54, v31, v47\nv_mfma_f32_16x16x32_bf16 v[244:247], v[68:71], v[60:63], v[244:247]\ns_waitcnt lgkmcnt(4)\n" \
  "v_mfma_f32_16x16x32_bf16 v[152:155], v[72:75], v[48:51], v[152:155]\nds_read_b64_tr_b16 v[68:69], v33 offset:49152\n" \
  "ds_read_b64_tr_b16 v[70:71], v33 offset:50176\nv_mfma_f32_16x16x32_bf16 v[184:187], v[72:75], v[52:55], v[184:187]\n" \
  "v_mfma_f32_16x16x32_bf16 v[216:219], v[72:75], v[56:59], v[216:219]\nv_mfma_f32_16x16x32_bf16 v[248:251], v[72:75], v[60:63], v[248:251]\n" \
  "s_waitcnt vmcnt(8)\ns_waitcnt lgkmcnt(4)\nv_mfma_f32_16x16x32_bf16 v[156:159], v[76:79], v[48:51], v[156:159]\n" \
  "ds_read_b64_tr_b16 v[72:73], v34 offset:49152\nds_read_b64_tr_b16 v[74:75], v34 offset:50176\nv_perm_b32 v48, v40, v104, s55\n" \
  "v_perm_b32 v49, v40, v104, s56\nv_perm_b32 v50, v41, v105, s55\nv_perm_b32 v51, v41, v105, s56\n" \
  "v_mfma_f32_16x16x32_bf16 v[188:191], v[76:79], v[52:55], v[188:191]\nv_perm_b32 v52, v42, v106, s55\nv_perm_b32 v53, v42, v106, s56\n" \
  "v_perm_b32 v54, v43, v107, s55\nv_perm_b32 v55, v43, v107, s56\nv_mfma_f32_16x16x32_bf16 v[220:223], v[76:79], v[56:59], v[220:223]\n" \
  "v_perm_b32 v56, v44, v108, s55\nv_perm_b32 v57, v44, v108, s56\nv_perm_b32 v58, v45, v109, s55\nv_perm_b32 v59, v45, v109, s56\n" \
  "v_mfma_f32_16x16x32_bf16 v[252:255], v[76:79], v[60:63], v[252:255]\nv_perm_b32 v60, v46, v110, s55\nv_perm_b32 v61, v46, v110, s56\n" \
  "v_perm_b32 v62, v47, v111, s55\nv_perm_b32 v63, v47, v111, s56\ns_bitcmp1_b32 s66, 6\ns_cbranch_scc1 LRAW5_%=\nLRET5_%=:\ns_waitcnt lgkmcnt(4)\n" \
  "v_mfma_f32_16x16x32_bf16 v[128:131], v[64:67], v[48:51], v[128:131]\nds_read_b64_tr_b16 v[76:77], v35 offset:49152\n" \
  "ds_read_b64_tr_b16 v[78:79], v35 offset:50176\nv_mfma_f32_16x16x32_bf16 v[160:163], v[64:67], v[52:55], v[160:163]\ns_bitcmp1_b32 s64, 8\n" \
  "s_cselect_b32 s60, 0x80, 0\nv_and_or_b32 v23, v17, s60, v16\nbuffer_load_dwordx3 v[88:90], v15, s[40:43], s58 offen\n" \
  "v_mfma_f32_16x16x32_bf16 v[192:195], v[64:67], v[56:59], v[192:195]\nbuffer_load_ushort v91, v23, s[40:43], s58 offen\n" \
  "v_mfma_f32_16x16x32_bf16 v[224:227], v[64:67], v[60:63], v[224:227]\ns_waitcnt vmcnt(6)\nv_bfi_b32 v22, s57, v127, v20\nv_lshrrev_b32 v31, 9, v127\n" \
  "s_waitcnt lgkmcnt(4)\nv_mfma_f32_16x16x32_bf16 v[132:135], v[68:71], v[48:51], v[132:135]\nds_read_b64_tr_b16 v[64:65], v36 offset:49152\n" \
  "ds_read_b64_tr_b16 v[66:67], v36 offset:50176\nv_mfma_f32_16x16x32_bf16 v[164:167], v[68:71], v[52:55], v[164:167]\nds_write_b8 v22, v31\n" \
  "ds_read_b128 v[40:43], v20\nv_mfma_f32_16x16x32_bf16 v[196:199], v[68:71], v[56:59], v[196:199]\n" \
  "buffer_load_dwordx4 v[80:83], v14, s[40:43], s58 offen\nds_read_b128 v[44:47], v20 offset:1024\nds_write_b8 v22, v21\n" \
  "v_mfma_f32_16x16x32_bf16 v[228:231], v[68:71], v[60:63], v[228:231]\nbuffer_load_dwordx4 v[84:87], v14, s[40:43], s58 offen offset:1024\n" \
  "s_add_u32 s58, s58, 47104\ns_waitcnt lgkmcnt(8)\nv_mfma_f32_16x16x32_bf16 v[136:139], v[72:75], v[48:51], v[136:139]\n" \
  "ds_read_b64_tr_b16 v[68:69], v37 offset:49152\nds_read_b64_tr_b16 v[70:71], v37 offset:50176\n" \
  "v_mfma_f32_16x16x32_bf16 v[168:171], v[72:75], v[52:55], v[168:171]\nv_alignbit_b32 v25, v124, v124, 3\n" \
  "v_mfma_f32_16x16x32_bf16 v[200:203], v[72:75], v[56:59], v[200:203]\nv_alignbit_b32 v26, v125, v125, 3\nv_alignbit_b32 v27, v126, v126, 3\n" \
  "v_mfma_f32_16x16x32_bf16 v[232:235], v[72:75], v[60:63], v[232:235]\nv_lshrrev_b32 v28, 5, v124\nv_lshrrev_b32 v29, 5, v125\ns_waitcnt lgkmcnt(8)\n" \
  "v_mfma_f32_16x16x32_bf16 v[140:143], v[76:79], v[48:51], v[140:143]\nds_read_b64_tr_b16 v[72:73], v38 offset:49152\n" \
  "ds_read_b64_tr_b16 v[74:75], v38 offset:50176\nv_mfma_f32_16x16x32_bf16 v[172:175], v[76:79], v[52:55], v[172:175]\nv_lshlrev_b32 v30, 2, v126\n" \
  "v_lshlrev_b32 v31, 1, v126\nv_mfma_f32_16x16x32_bf16 v[204:207], v[76:79], v[56:59], v[204:207]\ns_waitcnt lgkmcnt(5)\nv_bfi_b32 v40, s52, v124, v40\n" \
  "v_bfi_b32 v42, s52, v125, v42\nv_mfma_f32_16x16x32_bf16 v[236:239], v[76:79], v[60:63], v[236:239]\ns_add_u32 m0, s62, 16384\n" \
  "v_bfi_b32 v44, s52, v126, v44\nv_mfma_f32_16x16x32_bf16 v[144:147], v[64:67], v[48:51], v[144:147]\nds_read_b64_tr_b16 v[76:77], v39 offset:49152\n" \
  "ds_read_b64_tr_b16 v[78:79], v39 offset:50176\nv_mfma_f32_16x16x32_bf16 v[176:179], v[64:67], v[52:55], v[176:179]\n" \
  "buffer_load_dwordx4 v18, s[44:47], s59 offen offset:0 lds\nv_bfi_b32 v41, s52, v25, v41\nv_bfi_b32 v43, s52, v26, v43\n" \
  "v_mfma_f32_16x16x32_bf16 v[208:211], v[64:67], v[56:59], v[208:211]\nbuffer_load_dwordx4 v19, s[44:47], s59 offen offset:1024 lds\n" \
  "s_add_u32 s59, s59, 262144\nv_bfi_b32 v45, s52, v27, v45\nv_bfi_b32 v46, s53, v28, v46\n" \
  "v_mfma_f32_16x16x32_bf16 v[240:243], v[64:67], v[60:63], v[240:243]\ns_waitcnt lgkmcnt(4)\n" \
  "v_mfma_f32_16x16x32_bf16 v[148:151], v[68:71], v[48:51], v[148:151]\nds_read_b64_tr_b16 v[64:65], v32 offset:57344\n" \
  "ds_read_b64_tr_b16 v[66:67], v32 offset:58368\nv_mfma_f32_16x16x32_bf16 v[180:183], v[68:71], v[52:55], v[180:183]\nv_bfi_b32 v47, s53, v29, v47\n" \
  "v_bfi_b32 v46, s54, v30, v46\nv_mfma_f32_16x16x32_bf16 v[212:215], v[68:71], v[56:59], v[212:215]\nv_bfi_b32 v47, s54, v31, v47\n" \
  "v_mfma_f32_16x16x32_bf16 v[244:247], v[68:71], v[60:63], v[244:247]\ns_waitcnt lgkmcnt(4)\n" \
  "v_mfma_f32_16x16x32_bf16 v[152:155], v[72:75], v[48:51], v[152:155]\nds_read_b64_tr_b16 v[68:69], v33 offset:57344\n" \
  "ds_read_b64_tr_b16 v[70:71], v33 offset:58368\nv_mfma_f32_16x16x32_bf16 v[184:187], v[72:75], v[52:55], v[184:187]\n" \
  "v_mfma_f32_16x16x32_bf16 v[216:219], v[72:75], v[56:59], v[216:219]\nv_mfma_f32_16x16x32_bf16 v[248:251], v[72:75], v[60:63], v[248:251]\n" \
  "s_waitcnt vmcnt(8)\ns_waitcnt lgkmcnt(4)\nv_mfma_f32_16x16x32_bf16 v[156:159], v[76:79], v[48:51], v[156:159]\n" \
  "ds_read_b64_tr_b16 v[72:73], v34 offset:57344\nds_read_b64_tr_b16 v[74:75], v34 offset:58368\ns_setprio 2\nv_perm_b32 v48, v40, v116, s55\n" \
  "v_perm_b32 v49, v40, v116, s56\nv_perm_b32 v50, v41, v117, s55\nv_perm_b32 v51, v41, v117, s56\n" \
  "v_mfma_f32_16x16x32_bf16 v[188:191], v[76:79], v[52:55], v[188:191]\nv_perm_b32 v52, v42, v118, s55\nv_perm_b32 v53, v42, v118, s56\n" \
  "v_perm_b32 v54, v43, v119, s55\nv_perm_b32 v55, v43, v119, s56\nv_mfma_f32_16x16x32_bf16 v[220:223], v[76:79], v[56:59], v[220:223]\n" \
  "v_perm_b32 v56, v44, v120, s55\nv_perm_b32 v57, v44, v120, s56\nv_perm_b32 v58, v45, v121, s55\nv_perm_b32 v59, v45, v121, s56\n" \
  "v_mfma_f32_16x16x32_bf16 v[252:255], v[76:79], v[60:63], v[252:255]\nv_perm_b32 v60, v46, v122, s55\nv_perm_b32 v61, v46, v122, s56\n" \
  "v_perm_b32 v62, v47, v123, s55\nv_perm_b32 v63, v47, v123, s56\ns_bitcmp1_b32 s66, 7\ns_cbranch_scc1 LRAW6_%=\nLRET6_%=:\ns_cmp_eq_u32 s61, 7\n" \
  "s_cbranch_scc1 LLAST_%=\ns_waitcnt lgkmcnt(4)\nv_mfma_f32_16x16x32_bf16 v[128:131], v[64:67], v[48:51], v[128:131]\n" \
  "ds_read_b64_tr_b16 v[76:77], v35 offset:57344\nds_read_b64_tr_b16 v[78:79], v35 offset:58368\n" \
  "v_mfma_f32_16x16x32_bf16 v[160:163], v[64:67], v[52:55], v[160:163]\ns_bitcmp1_b32 s64, 9\ns_cselect_b32 s60, 0x80, 0\n" \
  "v_and_or_b32 v23, v17, s60, v16\nbuffer_load_dwordx3 v[100:102], v15, s[40:43], s58 offen\n" \
  "v_mfma_f32_16x16x32_bf16 v[192:195], v[64:67], v[56:59], v[192:195]\nbuffer_load_ushort v103, v23, s[40:43], s58 offen\n" \
  "v_mfma_f32_16x16x32_bf16 v[224:227], v[64:67], v[60:63], v[224:227]\ns_waitcnt vmcnt(6)\nv_bfi_b32 v22, s57, v91, v20\nv_lshrrev_b32 v31, 9, v91\n" \
  "s_waitcnt lgkmcnt(4)\nv_mfma_f32_16x16x32_bf16 v[132:135], v[68:71], v[48:51], v[132:135]\nds_read_b64_tr_b16 v[64:65], v36 offset:57344\n" \
  "ds_read_b64_tr_b16 v[66:67], v36 offset:58368\nv_mfma_f32_16x16x32_bf16 v[164:167], v[68:71], v[52:55], v[164:167]\nds_write_b8 v22, v31\n" \
  "ds_read_b128 v[40:43], v20\nv_mfma_f32_16x16x32_bf16 v[196:199], v[68:71], v[56:59], v[196:199]\n" \
  "buffer_load_dwordx4 v[92:95], v14, s[40:43], s58 offen\nds_read_b128 v[44:47], v20 offset:1024\nds_write_b8 v22, v21\n" \
  "v_mfma_f32_16x16x32_bf16 v[228:231], v[68:71], v[60:63], v[228:231]\nbuffer_load_dwordx4 v[96:99], v14, s[40:43], s58 offen offset:1024\n" \
  "s_add_u32 s58, s58, 47104\ns_waitcnt lgkmcnt(8)\nv_mfma_f32_16x16x32_bf16 v[136:139], v[72:75], v[48:51], v[136:139]\n" \
  "ds_read_b64_tr_b16 v[68:69], v37 offset:57344\nds_read_b64_tr_b16 v[70:71], v37 offset:58368\ns_setprio 1\n" \
  "v_mfma_f32_16x16x32_bf16 v[168:171], v[72:75], v[52:55], v[168:171]\nv_alignbit_b32 v25, v88, v88, 3\n" \
  "v_mfma_f32_16x16x32_bf16 v[200:203], v[72:75], v[56:59], v[200:203]\nv_alignbit_b32 v26, v89, v89, 3\nv_alignbit_b32 v27, v90, v90, 3\n" \
  "v_mfma_f32_16x16x32_bf16 v[232:235], v[72:75], v[60:63], v[232:235]\nv_lshrrev_b32 v28, 5, v88\nv_lshrrev_b32 v29, 5, v89\ns_waitcnt lgkmcnt(8)\n" \
  "v_mfma_f32_16x16x32_bf16 v[140:143], v[76:79], v[48:51], v[140:143]\nds_read_b64_tr_b16 v[72:73], v38 offset:57344\n" \
  "ds_read_b64_tr_b16 v[74:75], v38 offset:58368\nv_mfma_f32_16x16x32_bf16 v[172:175], v[76:79], v[52:55], v[172:175]\nv_lshlrev_b32 v30, 2, v90\n" \
  "v_lshlrev_b32 v31, 1, v90\nv_mfma_f32_16x16x32_bf16 v[204:207], v[76:79], v[56:59], v[204:207]\ns_waitcnt lgkmcnt(5)\nv_bfi_b32 v40, s52, v88, v40\n" \
  "v_bfi_b32 v42, s52, v89, v42\nv_mfma_f32_16x16x32_bf16 v[236:239], v[76:79], v[60:63], v[236:239]\ns_add_u32 m0, s62, 24576\n" \
  "v_bfi_b32 v44, s52, v90, v44\nv_mfma_f32_16x16x32_bf16 v[144:147], v[64:67], v[48:51], v[144:147]\nds_read_b64_tr_b16 v[76:77], v39 offset:57344\n" \
  "ds_read_b64_tr_b16 v[78:79], v39 offset:58368\ns_setprio 0\nv_mfma_f32_16x16x32_bf16 v[176:179], v[64:67], v[52:55], v[176:179]\n" \
  "buffer_load_dwordx4 v18, s[44:47], s59 offen offset:0 lds\nv_bfi_b32 v41, s52, v25, v41\nv_bfi_b32 v43, s52, v26, v43\n" \
  "v_mfma_f32_16x16x32_bf16 v[208:211], v[64:67], v[56:59], v[208:211]\nbuffer_load_dwordx4 v19, s[44:47], s59 offen offset:1024 lds\n" \
  "s_add_u32 s59, s59, 262144\nv_bfi_b32 v45, s52, v27, v45\nv_bfi_b32 v46, s53, v28, v46\n" \
  "v_mfma_f32_16x16x32_bf16 v[240:243], v[64:67], v[60:63], v[240:243]\ns_waitcnt lgkmcnt(4)\n" \
  "v_mfma_f32_16x16x32_bf16 v[148:151], v[68:71], v[48:51], v[148:151]\ns_barrier\nds_read_b64_tr_b16 v[64:65], v32 offset:0\n" \
  "ds_read_b64_tr_b16 v[66:67], v32 offset:1024\ns_setprio 3\nv_mfma_f32_16x16x32_bf16 v[180:183], v[68:71], v[52:55], v[180:183]\n" \
  "v_bfi_b32 v47, s53, v29, v47\nv_bfi_b32 v46, s54, v30, v46\nv_mfma_f32_16x16x32_bf16 v[212:215], v[68:71], v[56:59], v[212:215]\n" \
  "v_bfi_b32 v47, s54, v31, v47\nv_mfma_f32_16x16x32_bf16 v[244:247], v[68:71], v[60:63], v[244:247]\ns_waitcnt lgkmcnt(4)\n" \
  "v_mfma_f32_16x16x32_bf16 v[152:155], v[72:75], v[48:51], v[152:155]\nds_read_b64_tr_b16 v[68:69], v33 offset:0\n" \
  "ds_read_b64_tr_b16 v[70:71], v33 offset:1024\nv_mfma_f32_16x16x32_bf16 v[184:187], v[72:75], v[52:55], v[184:187]\n" \
  "v_mfma_f32_16x16x32_bf16 v[216:219], v[72:75], v[56:59], v[216:219]\nv_mfma_f32_16x16x32_bf16 v[248:251], v[72:75], v[60:63], v[248:251]\n" \
  "s_waitcnt vmcnt(8)\ns_waitcnt lgkmcnt(4)\nv_mfma_f32_16x16x32_bf16 v[156:159], v[76:79], v[48:51], v[156:159]\n" \
  "ds_read_b64_tr_b16 v[72:73], v34 offset:0\nds_read_b64_tr_b16 v[74:75], v34 offset:1024\nv_perm_b32 v48, v40, v80, s55\n" \
  "v_perm_b32 v49, v40, v80, s56\nv_perm_b32 v50, v41, v81, s55\nv_perm_b32 v51, v41, v81, s56\n" \
  "v_mfma_f32_16x16x32_bf16 v[188:191], v[76:79], v[52:55], v[188:191]\nv_perm_b32 v52, v42, v82, s55\nv_perm_b32 v53, v42, v82, s56\n" \
  "v_perm_b32 v54, v43, v83, s55\nv_perm_b32 v55, v43, v83, s56\nv_mfma_f32_16x16x32_bf16 v[220:223], v[76:79], v[56:59], v[220:223]\n" \
  "v_perm_b32 v56, v44, v84, s55\nv_perm_b32 v57, v44, v84, s56\nv_perm_b32 v58, v45, v85, s55\nv_perm_b32 v59, v45, v85, s56\n" \
  "v_mfma_f32_16x16x32_bf16 v[252:255], v[76:79], v[60:63], v[252:255]\nv_perm_b32 v60, v46, v86, s55\nv_perm_b32 v61, v46, v86, s56\n" \
  "v_perm_b32 v62, v47, v87, s55\nv_perm_b32 v63, v47, v87, s56\ns_bitcmp1_b32 s66, 8\ns_cbranch_scc1 LRAW7_%=\nLRET7_%=:\n" \
  "s_lshr_b64 s[64:65], s[64:65], 8\ns_lshr_b64 s[66:67], s[66:67], 8\ns_add_u32 s61, s61, 1\ns_branch LOOP_%=\nLRAWp_%=:\ns_lshr_b32 s60, s59, 2\n" \
  "s_add_u32 s60, s60, s63\nv_and_b32 v25, 0x3ff, v20\nbuffer_load_dwordx4 v[48:51], v25, s[48:51], s60 offen offset:0\n" \
  "buffer_load_dwordx4 v[52:55], v25, s[48:51], s60 offen offset:1024\nbuffer_load_dwordx4 v[56:59], v25, s[48:51], s60 offen offset:2048\n" \
  "buffer_load_dwordx4 v[60:63], v25, s[48:51], s60 offen offset:3072\ns_waitcnt vmcnt(0)\ns_branch LRETp_%=\nLRAW0_%=:\ns_lshr_b32 s60, s59, 2\n" \
  "s_add_u32 s60, s60, s63\nv_and_b32 v25, 0x3ff, v20\nbuffer_load_dwordx4 v[48:51], v25, s[48:51], s60 offen offset:0\n" \
  "buffer_load_dwordx4 v[52:55], v25, s[48:51], s60 offen offset:1024\nbuffer_load_dwordx4 v[56:59], v25, s[48:51], s60 offen offset:2048\n" \
  "buffer_load_dwordx4 v[60:63], v25, s[48:51], s60 offen offset:3072\ns_waitcnt vmcnt(0)\ns_branch LRET0_%=\nLRAW1_%=:\ns_lshr_b32 s60, s59, 2\n" \
  "s_add_u32 s60, s60, s63\nv_and_b32 v25, 0x3ff, v20\nbuffer_load_dwordx4 v[48:51], v25, s[48:51], s60 offen offset:0\n" \
  "buffer_load_dwordx4 v[52:55], v25, s[48:51], s60 offen offset:1024\nbuffer_load_dwordx4 v[56:59], v25, s[48:51], s60 offen offset:2048\n" \
  "buffer_load_dwordx4 v[60:63], v25, s[48:51], s60 offen offset:3072\ns_waitcnt vmcnt(0)\ns_branch LRET1_%=\nLRAW2_%=:\ns_lshr_b32 s60, s59, 2\n" \
  "s_add_u32 s60, s60, s63\nv_and_b32 v25, 0x3ff, v20\nbuffer_load_dwordx4 v[48:51], v25, s[48:51], s60 offen offset:0\n" \
  "buffer_load_dwordx4 v[52:55], v25, s[48:51], s60 offen offset:1024\nbuffer_load_dwordx4 v[56:59], v25, s[48:51], s60 offen offset:2048\n" \
  "buffer_load_dwordx4 v[60:63], v25, s[48:51], s60 offen offset:3072\ns_waitcnt vmcnt(0)\ns_branch LRET2_%=\nLRAW3_%=:\ns_lshr_b32 s60, s59, 2\n" \
  "s_add_u32 s60, s60, s63\nv_and_b32 v25, 0x3ff, v20\nbuffer_load_dwordx4 v[48:51], v25, s[48:51], s60 offen offset:0\n" \
  "buffer_load_dwordx4 v[52:55], v25, s[48:51], s60 offen offset:1024\nbuffer_load_dwordx4 v[56:59], v25, s[48:51], s60 offen offset:2048\n" \
  "buffer_load_dwordx4 v[60:63], v25, s[48:51], s60 offen offset:3072\ns_waitcnt vmcnt(0)\ns_branch LRET3_%=\nLRAW4_%=:\ns_lshr_b32 s60, s59, 2\n" \
  "s_add_u32 s60, s60, s63\nv_and_b32 v25, 0x3ff, v20\nbuffer_load_dwordx4 v[48:51], v25, s[48:51], s60 offen offset:0\n" \
  "buffer_load_dwordx4 v[52:55], v25, s[48:51], s60 offen offset:1024\nbuffer_load_dwordx4 v[56:59], v25, s[48:51], s60 offen offset:2048\n" \
  "buffer_load_dwordx4 v[60:63], v25, s[48:51], s60 offen offset:3072\ns_waitcnt vmcnt(0)\ns_branch LRET4_%=\nLRAW5_%=:\ns_lshr_b32 s60, s59, 2\n" \
  "s_add_u32 s60, s60, s63\nv_and_b32 v25, 0x3ff, v20\nbuffer_load_dwordx4 v[48:51], v25, s[48:51], s60 offen offset:0\n" \
  "buffer_load_dwordx4 v[52:55], v25, s[48:51], s60 offen offset:1024\nbuffer_load_dwordx4 v[56:59], v25, s[48:51], s60 offen offset:2048\n" \
  "buffer_load_dwordx4 v[60:63], v25, s[48:51], s60 offen offset:3072\ns_waitcnt vmcnt(0)\ns_branch LRET5_%=\nLRAW6_%=:\ns_lshr_b32 s60, s59, 2\n" \
  "s_add_u32 s60, s60, s63\nv_and_b32 v25, 0x3ff, v20\nbuffer_load_dwordx4 v[48:51], v25, s[48:51], s60 offen offset:0\n" \
  "buffer_load_dwordx4 v[52:55], v25, s[48:51], s60 offen offset:1024\nbuffer_load_dwordx4 v[56:59], v25, s[48:51], s60 offen offset:2048\n" \
  "buffer_load_dwordx4 v[60:63], v25, s[48:51], s60 offen offset:3072\ns_waitcnt vmcnt(0)\ns_branch LRET6_%=\nLRAW7_%=:\ns_lshr_b32 s60, s59, 2\n" \
  "s_add_u32 s60, s60, s63\nv_and_b32 v25, 0x3ff, v20\nbuffer_load_dwordx4 v[48:51], v25, s[48:51], s60 offen offset:0\n" \
  "buffer_load_dwordx4 v[52:55], v25, s[48:51], s60 offen offset:1024\nbuffer_load_dwordx4 v[56:59], v25, s[48:51], s60 offen offset:2048\n" \
  "buffer_load_dwordx4 v[60:63], v25, s[48:51], s60 offen offset:3072\ns_waitcnt vmcnt(0)\ns_branch LRET7_%=\nLLAST_%=:\nv_and_b32 v25, 0x3ff, v20\n" \
  "v_add_u32 v25, s62, v25\nv_xor_b32 v26, 0x10000, v25\nv_mov_b32 v12, 0\ns_waitcnt vmcnt(0)\ns_waitcnt lgkmcnt(4)\n" \
  "v_mfma_f32_16x16x32_bf16 v[128:131], v[64:67], v[48:51], v[128:131]\nds_read_b64_tr_b16 v[76:77], v35 offset:57344\n" \
  "ds_read_b64_tr_b16 v[78:79], v35 offset:58368\nv_mfma_f32_16x16x32_bf16 v[160:163], v[64:67], v[52:55], v[160:163]\n" \
  "v_mfma_f32_16x16x32_bf16 v[192:195], v[64:67], v[56:59], v[192:195]\nv_mfma_f32_16x16x32_bf16 v[224:227], v[64:67], v[60:63], v[224:227]\n" \
  "s_waitcnt lgkmcnt(4)\nv_mfma_f32_16x16x32_bf16 v[132:135], v[68:71], v[48:51], v[132:135]\nds_read_b64_tr_b16 v[64:65], v36 offset:57344\n" \
  "ds_read_b64_tr_b16 v[66:67], v36 offset:58368\nv_mfma_f32_16x16x32_bf16 v[164:167], v[68:71], v[52:55], v[164:167]\n" \
  "v_mfma_f32_16x16x32_bf16 v[196:199], v[68:71], v[56:59], v[196:199]\nv_mfma_f32_16x16x32_bf16 v[228:231], v[68:71], v[60:63], v[228:231]\n" \
  "s_waitcnt lgkmcnt(4)\nv_mfma_f32_16x16x32_bf16 v[136:139], v[72:75], v[48:51], v[136:139]\nds_read_b64_tr_b16 v[68:69], v37 offset:57344\n" \
  "ds_read_b64_tr_b16 v[70:71], v37 offset:58368\nv_mfma_f32_16x16x32_bf16 v[168:171], v[72:75], v[52:55], v[168:171]\n" \
  "v_mfma_f32_16x16x32_bf16 v[200:203], v[72:75], v[56:59], v[200:203]\nds_write_b128 v25, v[128:131] offset:0\n" \
  "v_mfma_f32_16x16x32_bf16 v[232:235], v[72:75], v[60:63], v[232:235]\nds_write_b128 v25, v[160:163] offset:1024\ns_waitcnt lgkmcnt(6)\n" \
  "v_mfma_f32_16x16x32_bf16 v[140:143], v[76:79], v[48:51], v[140:143]\nds_read_b64_tr_b16 v[72:73], v38 offset:57344\n" \
  "ds_read_b64_tr_b16 v[74:75], v38 offset:58368\nv_mfma_f32_16x16x32_bf16 v[172:175], v[76:79], v[52:55], v[172:175]\n" \
  "v_mfma_f32_16x16x32_bf16 v[204:207], v[76:79], v[56:59], v[204:207]\nds_write_b128 v25, v[132:135] offset:8192\n" \
  "v_mfma_f32_16x16x32_bf16 v[236:239], v[76:79], v[60:63], v[236:239]\nds_write_b128 v25, v[164:167] offset:9216\ns_waitcnt lgkmcnt(8)\n" \
  "v_mfma_f32_16x16x32_bf16 v[144:147], v[64:67], v[48:51], v[144:147]\nds_read_b64_tr_b16 v[76:77], v39 offset:57344\n" \
  "ds_read_b64_tr_b16 v[78:79], v39 offset:58368\nv_mfma_f32_16x16x32_bf16 v[176:179], v[64:67], v[52:55], v[176:179]\n" \
  "v_mfma_f32_16x16x32_bf16 v[208:211], v[64:67], v[56:59], v[208:211]\nds_write_b128 v25, v[136:139] offset:16384\n" \
  "v_mfma_f32_16x16x32_bf16 v[240:243], v[64:67], v[60:63], v[240:243]\nds_write_b128 v25, v[168:171] offset:17408\ns_waitcnt lgkmcnt(10)\n" \
  "s_waitcnt lgkmcnt(2)\ns_barrier\nv_mfma_f32_16x16x32_bf16 v[148:151], v[68:71], v[48:51], v[148:151]\n" \
  "v_mfma_f32_16x16x32_bf16 v[180:183], v[68:71], v[52:55], v[180:183]\nv_mfma_f32_16x16x32_bf16 v[212:215], v[68:71], v[56:59], v[212:215]\n" \
  "ds_write_b128 v25, v[140:143] offset:24576\nv_mfma_f32_16x16x32_bf16 v[244:247], v[68:71], v[60:63], v[244:247]\n" \
  "ds_write_b128 v25, v[172:175] offset:25600\nv_mfma_f32_16x16x32_bf16 v[152:155], v[72:75], v[48:51], v[152:155]\n" \
  "v_mfma_f32_16x16x32_bf16 v[184:187], v[72:75], v[52:55], v[184:187]\nv_mfma_f32_16x16x32_bf16 v[216:219], v[72:75], v[56:59], v[216:219]\n" \
  "ds_write_b128 v25, v[144:147] offset:32768\nv_mfma_f32_16x16x32_bf16 v[248:251], v[72:75], v[60:63], v[248:251]\n" \
  "ds_write_b128 v25, v[176:179] offset:33792\nv_mfma_f32_16x16x32_bf16 v[156:159], v[76:79], v[48:51], v[156:159]\n" \
  "v_mfma_f32_16x16x32_bf16 v[188:191], v[76:79], v[52:55], v[188:191]\nv_mfma_f32_16x16x32_bf16 v[220:223], v[76:79], v[56:59], v[220:223]\n" \
  "ds_write_b128 v25, v[148:151] offset:40960\nv_mfma_f32_16x16x32_bf16 v[252:255], v[76:79], v[60:63], v[252:255]\n" \
  "ds_write_b128 v25, v[180:183] offset:41984\ns_nop 2\nds_write_b128 v25, v[152:155] offset:49152\nds_write_b128 v25, v[184:187] offset:50176\ns_nop 3\n" \
  "ds_write_b128 v25, v[156:159] offset:57344\nds_write_b128 v25, v[188:191] offset:58368\ns_waitcnt lgkmcnt(0)\ns_barrier\n" \
  "ds_read_b128 v[128:131], v26 offset:0\nds_read_b128 v[160:163], v26 offset:1024\nds_read_b128 v[132:135], v26 offset:8192\n" \
  "ds_read_b128 v[164:167], v26 offset:9216\nds_read_b128 v[136:139], v26 offset:16384\nds_read_b128 v[168:171], v26 offset:17408\n" \
  "ds_read_b128 v[140:143], v26 offset:24576\nds_read_b128 v[172:175], v26 offset:25600\nds_read_b128 v[144:147], v26 offset:32768\n" \
  "ds_read_b128 v[176:179], v26 offset:33792\nds_read_b128 v[148:151], v26 offset:40960\nds_read_b128 v[180:183], v26 offset:41984\n" \
  "ds_read_b128 v[152:155], v26 offset:49152\nds_read_b128 v[184:187], v26 offset:50176\nds_read_b128 v[156:159], v26 offset:57344\n" \
  "ds_read_b128 v[188:191], v26 offset:58368\ns_waitcnt lgkmcnt(15)\nv_add_f32 v192, v192, v128\nv_add_f32 v193, v193, v129\nv_add_f32 v194, v194, v130\n" \
  "v_add_f32 v195, v195, v131\nv_max3_f32 v12, |v192|, |v193|, v12\nv_max3_f32 v12, |v194|, |v195|, v12\ns_waitcnt lgkmcnt(14)\n" \
  "v_add_f32 v224, v224, v160\nv_add_f32 v225, v225, v161\nv_add_f32 v226, v226, v162\nv_add_f32 v227, v227, v163\nv_max3_f32 v12, |v224|, |v225|, v12\n" \
  "v_max3_f32 v12, |v226|, |v227|, v12\ns_waitcnt lgkmcnt(13)\nv_add_f32 v196, v196, v132\nv_add_f32 v197, v197, v133\nv_add_f32 v198, v198, v134\n" \
  "v_add_f32 v199, v199, v135\nv_max3_f32 v12, |v196|, |v197|, v12\nv_max3_f32 v12, |v198|, |v199|, v12\ns_waitcnt lgkmcnt(12)\n" \
  "v_add_f32 v228, v228, v164\nv_add_f32 v229, v229, v165\nv_add_f32 v230, v230, v166\nv_add_f32 v231, v231, v167\nv_max3_f32 v12, |v228|, |v229|, v12\n" \
  "v_max3_f32 v12, |v230|, |v231|, v12\ns_waitcnt lgkmcnt(11)\nv_add_f32 v200, v200, v136\nv_add_f32 v201, v201, v137\nv_add_f32 v202, v202, v138\n" \
  "v_add_f32 v203, v203, v139\nv_max3_f32 v12, |v200|, |v201|, v12\nv_max3_f32 v12, |v202|, |v203|, v12\ns_waitcnt lgkmcnt(10)\n" \
  "v_add_f32 v232, v232, v168\nv_add_f32 v233, v233, v169\nv_add_f32 v234, v234, v170\nv_add_f32 v235, v235, v171\nv_max3_f32 v12, |v232|, |v233|, v12\n" \
  "v_max3_f32 v12, |v234|, |v235|, v12\ns_waitcnt lgkmcnt(9)\nv_add_f32 v204, v204, v140\nv_add_f32 v205, v205, v141\nv_add_f32 v206, v206, v142\n" \
  "v_add_f32 v207, v207, v143\nv_max3_f32 v12, |v204|, |v205|, v12\nv_max3_f32 v12, |v206|, |v207|, v12\ns_waitcnt lgkmcnt(8)\n" \
  "v_add_f32 v236, v236, v172\nv_add_f32 v237, v237, v173\nv_add_f32 v238, v238, v174\nv_add_f32 v239, v239, v175\nv_max3_f32 v12, |v236|, |v237|, v12\n" \
  "v_max3_f32 v12, |v238|, |v239|, v12\ns_waitcnt lgkmcnt(7)\nv_add_f32 v208, v208, v144\nv_add_f32 v209, v209, v145\nv_add_f32 v210, v210, v146\n" \
  "v_add_f32 v211, v211, v147\nv_max3_f32 v12, |v208|, |v209|, v12\nv_max3_f32 v12, |v210|, |v211|, v12\ns_waitcnt lgkmcnt(6)\n" \
  "v_add_f32 v240, v240, v176\nv_add_f32 v241, v241, v177\nv_add_f32 v242, v242, v178\nv_add_f32 v243, v243, v179\nv_max3_f32 v12, |v240|, |v241|, v12\n" \
  "v_max3_f32 v12, |v242|, |v243|, v12\ns_waitcnt lgkmcnt(5)\nv_add_f32 v212, v212, v148\nv_add_f32 v213, v213, v149\nv_add_f32 v214, v214, v150\n" \
  "v_add_f32 v215, v215, v151\nv_max3_f32 v12, |v212|, |v213|, v12\nv_max3_f32 v12, |v214|, |v215|, v12\ns_waitcnt lgkmcnt(4)\n" \
  "v_add_f32 v244, v244, v180\nv_add_f32 v245, v245, v181\nv_add_f32 v246, v246, v182\nv_add_f32 v247, v247, v183\nv_max3_f32 v12, |v244|, |v245|, v12\n" \
  "v_max3_f32 v12, |v246|, |v247|, v12\ns_waitcnt lgkmcnt(3)\nv_add_f32 v216, v216, v152\nv_add_f32 v217, v217, v153\nv_add_f32 v218, v218, v154\n" \
  "v_add_f32 v219, v219, v155\nv_max3_f32 v12, |v216|, |v217|, v12\nv_max3_f32 v12, |v218|, |v219|, v12\ns_waitcnt lgkmcnt(2)\n" \
  "v_add_f32 v248, v248, v184\nv_add_f32 v249, v249, v185\nv_add_f32 v250, v250, v186\nv_add_f32 v251, v251, v187\nv_max3_f32 v12, |v248|, |v249|, v12\n" \
  "v_max3_f32 v12, |v250|, |v251|, v12\ns_waitcnt lgkmcnt(1)\nv_add_f32 v220, v220, v156\nv_add_f32 v221, v221, v157\nv_add_f32 v222, v222, v158\n" \
  "v_add_f32 v223, v223, v159\nv_max3_f32 v12, |v220|, |v221|, v12\nv_max3_f32 v12, |v222|, |v223|, v12\ns_waitcnt lgkmcnt(0)\n" \
  "v_add_f32 v252, v252, v188\nv_add_f32 v253, v253, v189\nv_add_f32 v254, v254, v190\nv_add_f32 v255, v255, v191\nv_max3_f32 v12, |v252|, |v253|, v12\n" \
  "v_max3_f32 v12, |v254|, |v255|, v12\n"

__global__ void __launch_bounds__(512, 2) ag_gemm_q(Params p) {
  __shared__ __attribute__((aligned(2048))) char smem_raw[L_TOTAL];
  LDSP char* smem = (LDSP char*)smem_raw;
  const u32 tid = threadIdx.x, lane = tid & 63;
  const u32 v = rfl(tid >> 6);
  const u32 b = blockIdx.x;
  const u32 rank = p.rank & 7;
  const u32 hh = (b >> 3) & 1, c = b >> 4;
  char* const myws = p.myws;
  char* outp = p.out;
  const char* xin = p.x;
  asm volatile("" : "+s"(xin));
  auto peer_of_lane = [&]() -> char* {
    u32 lo = 0, hi = 0;
#pragma unroll
    for (int k = 0; k < NR; ++k) {
      asm("v_writelane_b32 %0, %1, %2" : "+v"(lo) : "s"((u32)(u64)p.ws[k]), "n"(k));
      asm("v_writelane_b32 %0, %1, %2" : "+v"(hi) : "s"((u32)((u64)p.ws[k] >> 32)), "n"(k));
    }
    return (char*)(((u64)hi << 32) | lo);
  };
  const u64 t0 = now();
  GLBP u32* const cnt_ptr = (GLBP u32*)(myws + CNT_OFF + b * 64);
  GLBP u32* const tmo_ptr = (GLBP u32*)(myws + TMO_OFF);
  LDSP u32* red = (LDSP u32*)(smem + L_RED);
  STAMP(0);
  const u32x4 srdw = make_srd(p.w, W_BYTES);
  u32 seq;

  if (v < 4) {
    const u32 est = b >> 2, mbi = (b & 3) * 4 + v, ewb = mbi >> 1, mb = mbi & 1;
    u32x4 xv[4], xs;
    {
      xs = *(const GLBP u32x4*)(xin + (size_t)(ewb * 64 + lane) * (KD * 2));
      const char* xp = xin + (size_t)(ewb * 64 + mb * 32 + (lane & 15)) * (KD * 2) + est * 128 + mb * 64 + (lane >> 4) * 16;
      const char* xq = xp + (mb ? -32 : 32) * (KD * 2);
      xv[0] = *(const GLBP u32x4*)xp; xv[1] = *(const GLBP u32x4*)(xp + 16 * (KD * 2));
      xv[2] = *(const GLBP u32x4*)xq; xv[3] = *(const GLBP u32x4*)(xq + 16 * (KD * 2));
    }
    const u32 cntv = __hip_atomic_load(cnt_ptr, __ATOMIC_RELAXED, __HIP_MEMORY_SCOPE_SYSTEM);
    char* const peer = peer_of_lane();
    LDSP char* hp = smem + L_ENC + v * 2304;
    LDSP unsigned short* list = (LDSP unsigned short*)(hp + 2048);
    list[lane] = (unsigned short)(lane * 16);
    u32 sum = __builtin_amdgcn_sad_u8(__builtin_amdgcn_perm(xs[1] << 1, xs[0] << 1, 0x07050301u), 0, 0);
    sum = __builtin_amdgcn_sad_u8(__builtin_amdgcn_perm(xs[3] << 1, xs[2] << 1, 0x07050301u), 0, sum);
    int ebi = (int)((last_lane(wave_scan(sum)) + 932u + 511u) >> 9) - 7;
    ebi = ebi < 0 ? 0 : (ebi > 248 ? 248 : ebi);
    const u32 kk = (u32)(ebi < 64 ? ebi + 7 : ebi) >> 3;
    const u32 up = (u32)((int)kk * 8 - ebi + 7);
    const float sc = __builtin_bit_cast(float, (120u + up) << 23);
    STAMP(1);
    u32 L[8], H[8];
    u32 mask = 0;
#pragma unroll
    for (int q = 0; q < 16; ++q) {
      const u32 t = xv[q >> 2][q & 3];
      const float lo = __builtin_bit_cast(float, t << 16) * sc, hi = __builtin_bit_cast(float, t & 0xffff0000u) * sc;
      xv[q >> 2][q & 3] = __builtin_amdgcn_perm(__builtin_bit_cast(u32, hi), __builtin_bit_cast(u32, lo), 0x07060302u);
    }
#pragma unroll
    for (int q = 0; q < 8; ++q) {
      const u32 y0 = xv[q >> 1][2 * (q & 1)], y1 = xv[q >> 1][2 * (q & 1) + 1];
      L[q] = __builtin_amdgcn_perm(y1, y0, 0x06040200u);
      H[q] = __builtin_amdgcn_perm(y1, y0, 0x07050301u);
    }
    __amdgpu_buffer_rsrc_t rc = __builtin_amdgcn_make_buffer_rsrc(myws + XC_OFF, 0, (int)XC_BYTES, 0x00020000);
    const u32 blk = est * STEP_BYTES + ewb * WB_BYTES;
    {
      u32x4 l0 = {L[0], L[1], L[2], L[3]}, l1 = {L[4], L[5], L[6], L[7]};
      __builtin_amdgcn_raw_buffer_store_b128(l0, rc, lane * 16, blk + mb * 2048, 17);
      __builtin_amdgcn_raw_buffer_store_b128(l1, rc, lane * 16 + 1024, blk + mb * 2048, 17);
      u32x4 h0 = {H[0], H[1], H[2], H[3]}, h1 = {H[4], H[5], H[6], H[7]};
      *(LDSP u32x4*)(hp + lane * 16) = h0;
      *(LDSP u32x4*)(hp + 1024 + lane * 16) = h1;
    }
#pragma unroll
    for (int q = 0; q < 8; ++q)
      mask |= ((((H[q] ^ (kk * 0x04040404u)) & 0x7c7c7c7cu) + 0x7c7c7c7cu) >> (7 - q)) & (0x01010101u << q);
    const u32 mine = (u32)__builtin_popcount(mask);
    u32 slot = wave_scan(mine);
    const u32 cnt = last_lane(slot);
    slot -= mine;
    const bool raw = CFG_FORCE_RAW || cnt > 64;
    if (!raw) {
      while (mask) {
        const u32 bit = (u32)__builtin_ctz(mask);
        list[slot++] = (unsigned short)(((bit & 4) << 8) + lane * 16 + (bit & 3) * 4 + (bit >> 3));
        mask &= mask - 1;
      }
    }
    {
      const u32 M = 0x83838383u;
      u32x3 cw;
      cw[0] = (H[0] & M) | __builtin_amdgcn_alignbit(H[1] & M, H[1] & M, 29) | ((H[6] & 0x03030303u) << 5);
      cw[1] = (H[2] & M) | __builtin_amdgcn_alignbit(H[3] & M, H[3] & M, 29) | ((H[7] & 0x03030303u) << 5);
      cw[2] = (H[4] & M) | __builtin_amdgcn_alignbit(H[5] & M, H[5] & M, 29) | ((H[6] & 0x80808080u) >> 2) | ((H[7] & 0x80808080u) >> 1);
      __builtin_amdgcn_raw_buffer_store_b96(cw, rc, lane * 12, blk + 4096 + mb * 768, 17);
    }
    {
      asm volatile("" ::: "memory");
      const u32 idx = list[lane];
      const u32 hb = *(LDSP unsigned char*)(hp + idx);
      if (lane < 32 || cnt > 32)
        __builtin_amdgcn_raw_buffer_store_b16((unsigned short)(((hb & 0x7cu) << 9) | idx), rc,
                                              (lane >> 5) * 128 + (lane & 31) * 2, blk + 5632 + mb * 64, 17);
    }
    if (raw) {
      __amdgpu_buffer_rsrc_t rr = __builtin_amdgcn_make_buffer_rsrc(myws + RAW_OFF, 0, (int)RAW_BYTES, 0x00020000);
#pragma unroll
      for (int s = 0; s < 4; ++s)
        __builtin_amdgcn_raw_buffer_store_b128(xv[s], rr, lane * 16 + s * 1024, (est * 16 + mbi) * 4096, 17);
    }
    seq = rfl(cntv) + 1;
    STAMP(2);
    asm volatile("s_waitcnt vmcnt(0)" ::: "memory");
    if (lane < NR) {
      u32x2 gr = {seq, kk | (up << 16) | (raw ? 0x200u : (cnt > 32 ? 0x100u : 0u))};
      char* fp = peer + XF_OFF + rank * 16 * XF_B + (est < 2 ? 2 * ewb * XF_B + 512 + mb * 16 : mbi * XF_B) + est * 8;
      asm volatile("global_store_dwordx2 %0, %1, off sc0 sc1" :: "v"(fp), "v"(gr) : "memory");
    }
    STAMP(3);
  } else {
    const u32 xcc = __builtin_amdgcn_s_getreg((31 << 11) | 20) & 7u;
    if (v == 7) {
      if (b < NR) asm volatile("buffer_inv sc0 sc1" ::: "memory");
      else asm volatile("buffer_inv sc0" ::: "memory");
    }
    u32 cntv;
    asm volatile("global_load_dword %0, %1, off sc0 sc1" : "=&v"(cntv) : "v"(cnt_ptr) : "memory");
    auto wpre = [&]()
    {
      const u32 rq = lane >> 4;
#pragma unroll
      for (int h2 = 0; h2 < 2; ++h2) {
        const u32 vv = (v - 4) * 2 + h2;
        const u32 o0 = (8 * vv + rq) * 4096 + c * 256 + (((lane & 15) ^ (rq << 1) ^ (h2 << 3)) * 16);
        const u32 o1 = o0 + 16384 - 1024;
#pragma unroll
        for (int d = 0; d < MAIN_NP; ++d)
          asm volatile("s_mov_b32 m0, %4\n\ts_nop 0\n\tbuffer_load_dwordx4 %0, %2, %3 offen lds\n\t"
                       "buffer_load_dwordx4 %1, %2, %3 offen offset:1024 lds"
                       :: "v"(o0), "v"(o1), "s"(srdw), "s"(d * 64 * 4096), "s"(L_W + (vv >> 2) * 65536 + d * 8192 + (vv & 3) * 2048) : "memory");
      }
    };
    if (tid < 256 + 16) red[lane] = 0;
    asm volatile("s_waitcnt vmcnt(0)" : "+v"(cntv) :: "memory");
    seq = rfl(cntv) + 1;
    STAMP(1);
    if (v == 7) {
      char* ip = myws + INV_OFF + (b & 63) * 128;
      const u32x2 gr = {seq, xcc};
      if (b < NR && lane < 8) asm volatile("global_store_dwordx2 %0, %1, off sc0 sc1" :: "v"(ip + lane * 1024), "v"(gr) : "memory");
      if (b == 0) asm volatile("global_store_dwordx2 %0, %1, off sc0 sc1" :: "v"(ip + 8192 + lane * 128), "v"(gr) : "memory");
      if (lane) ip += 8192;
      u32x2 f;
      for (;;) {
        asm volatile("global_load_dwordx2 %0, %1, off sc0 sc1\n\ts_waitcnt vmcnt(0)" : "=&v"(f) : "v"(ip) : "memory");
        if (__builtin_amdgcn_ballot_w64(f[0] < seq) == 0) break;
        if (expired(t0, tmo_ptr, seq)) { f[1] = 8; break; }
        if (INV_SLP) __builtin_amdgcn_s_sleep(INV_SLP);
      }
      if (rfl(f[1]) != xcc)
        asm volatile("buffer_inv sc0 sc1\n\ts_waitcnt vmcnt(0)" ::: "memory");
      if (lane == 1) red[16] = f[1];
    }
    STAMP(2);
    wpre();
  }
  wg_barrier();
  STAMP(4);

  const u32 rb = 4 * hh + (v & 3), kh = v >> 2, mbi = 2 * rb + kh;
  const u32 xcc0 = rfl(red[16]);
  const u32 xi = (b + xcc0) & 7, g = (p.order >> (4 * xi)) & 7;
  const char* const srcws = p.ws[xi];
  u32 meta;
  bool ok = true;
  if (xcc0 > 7 && lane == 0) __hip_atomic_store(&red[2], 1u, __ATOMIC_RELAXED, __HIP_MEMORY_SCOPE_WORKGROUP);
  {
    const char* const fb = myws + XF_OFF + g * 16 * XF_B;
    const char* const sp = fb + (mbi & ~1u) * XF_B + 512 + (lane & 3) * 8;
    const char* const fp = lane < 2 ? sp + kh * 16 : fb + mbi * XF_B + lane * 8;
    u32x2 f = {0, 0};
    for (;;) {
      u32 s4;
      asm volatile("global_load_dword %0, %1, off sc0 sc1\n\ts_waitcnt vmcnt(0)" : "=&v"(s4) : "v"(sp) : "memory");
      if (__builtin_amdgcn_ballot_w64(s4 < seq) == 0) break;
      if (expired(t0, tmo_ptr, seq)) { ok = false; break; }
    }
    if (ok && (c & 7) == 1) {
      const u32 li = lane >> 1;
      const char* tp = srcws + XC_OFF + rb * WB_BYTES + (lane & 1) * 64 +
                       (li < 16 ? kh * 2048 + li * 128 : (li < 22 ? 4096 + kh * 768 + (li - 16) * 128 : 5632));
      asm volatile("global_load_dwordx2 %0, %1, off sc0 sc1\n\ts_mov_b32 m0, %3\n\ts_nop 0\n\t"
                   "global_load_lds_dword %2, off\n\ts_waitcnt vmcnt(1)" : "=&v"(f) : "v"(fp), "v"(tp), "s"(L_DUMP) : "memory");
    }
    while (ok && __builtin_amdgcn_ballot_w64(f[0] < seq) != 0) {
      if (f[0] < seq)
        asm volatile("global_load_dwordx2 %0, %1, off sc0 sc1\n\ts_waitcnt vmcnt(0)" : "+&v"(f) : "v"(fp) : "memory");
      if (__builtin_amdgcn_ballot_w64(f[0] < seq) == 0) break;
      if (expired(t0, tmo_ptr, seq)) ok = false;
    }
    if (!ok && lane == 0) __hip_atomic_store(&red[2], 1u, __ATOMIC_RELAXED, __HIP_MEMORY_SCOPE_WORKGROUP);
    meta = ok ? f[1] : 0u;
  }
  STAMP(5);
  asm volatile("" : "+s"(outp));
  f32x16 acc0, acc1, acc2, acc3;
  u32 am;
  const u32 meta0 = rfl(meta);
  {
    u32x4 sx = make_srd(srcws + XC_OFF, ok ? XC_BYTES : 0u), sw = srdw;
    const u32x4 sr = make_srd(srcws + RAW_OFF, ok ? RAW_BYTES : 0u);
    u64 ovm = __builtin_amdgcn_ballot_w64((meta & 0x100u) != 0), rawm = __builtin_amdgcn_ballot_w64((meta & 0x200u) != 0);
    const u32 wbo = (mbi >> 1) * WB_BYTES, mb = mbi & 1;
    const u32 rq = lane >> 4;
    u32x8 wa;
    {
      const u32 q = (lane >> 2) & 3;
#pragma unroll
      for (int nb = 0; nb < 8; ++nb)
        wa[nb] = L_W + kh * 65536 + (8 * rq + q) * 256 + ((nb ^ q ^ ((rq & 1) << 2)) * 32) + 8 * (lane & 3);
    }
    const u32 vw0 = (8 * v + rq) * 4096 + c * 256 + (((lane & 15) ^ (rq << 1) ^ ((v & 1) << 3)) * 16);
    asm volatile(MAIN_ASM
                 : "={v12}"(am), "={v[192:207]}"(acc0), "={v[208:223]}"(acc1), "={v[224:239]}"(acc2), "={v[240:255]}"(acc3),
                   "+{s[40:43]}"(sx), "+{s[44:47]}"(sw), "+{s[64:65]}"(ovm), "+{s[66:67]}"(rawm)
                 : "{s[48:51]}"(sr), "{s62}"(L_W + kh * 65536 + (v & 3) * 2048), "{s63}"(mbi * 4096 - MAIN_WD * 65536), "{v[32:39]}"(wa),
                   "{v14}"(wbo + mb * 2048 + lane * 16), "{v15}"(wbo + 4096 + mb * 768 + lane * 12),
                   "{v16}"(wbo + 5632 + mb * 64 + (lane & 31) * 2), "{v17}"((lane >> 5) * 128),
                   "{v18}"(vw0), "{v19}"(vw0 + 16384 - 1024), "{v20}"(L_PL + v * 2048 + lane * 16),
                   "{v21}"((meta0 & 31u) * 0x04040404u)
                 : MAIN_CLOB, "scc", "memory");
  }
  STAMP(6);
  u32 ln;
  asm volatile("v_mbcnt_lo_u32_b32 %0, -1, 0\n\tv_mbcnt_hi_u32_b32 %0, -1, %0" : "=v"(ln));
  const float unsc = __builtin_bit_cast(float, (134u - ((meta0 >> 16) & 15u)) << 23);
  am = __builtin_bit_cast(u32, __builtin_bit_cast(float, wave_umax(am)) * unsc);
  u32 ticket = 0;
  if (ln == 0) {
    __hip_atomic_fetch_max(&red[0], am, __ATOMIC_RELAXED, __HIP_MEMORY_SCOPE_WORKGROUP);
    ticket = __hip_atomic_fetch_add(&red[1], 1u, __ATOMIC_RELAXED, __HIP_MEMORY_SCOPE_WORKGROUP);
  }
  const bool closer = rfl(ticket) == 7;
  if (closer) {
    u32 wm = __hip_atomic_load(&red[0], __ATOMIC_RELAXED, __HIP_MEMORY_SCOPE_WORKGROUP);
    if (__hip_atomic_load(&red[2], __ATOMIC_RELAXED, __HIP_MEMORY_SCOPE_WORKGROUP)) wm = 0x7fffffffu;
    if (ln == 0) {
      u32x2 gr = {seq, wm};
      asm volatile("global_store_dwordx2 %0, %1, off sc0 sc1" :: "v"(myws + AGL_OFF + b * 8), "v"(gr) : "memory");
    }
  }
  STAMP(7);
  if (b < NR && xi == SELF_XCD && v < CFG_NSW) {
    char* const peer = *(char* const GLBP*)((const GLBP char*)__builtin_amdgcn_kernarg_segment_ptr() + (ln & 7) * 8);
    const char* gp = myws + AGL_OFF + ln * 16;
    u32 gm = 0;
    bool done = false;
    for (;;) {
      if (!done) {
        u32x4 a0, a1;
        asm volatile("global_load_dwordx4 %0, %2, off sc0 sc1\n\tglobal_load_dwordx4 %1, %2, off offset:1024 sc0 sc1\n\t"
                     "s_waitcnt vmcnt(0)" : "=&v"(a0), "=&v"(a1) : "v"(gp) : "memory");
        done = a0[0] >= seq && a0[2] >= seq && a1[0] >= seq && a1[2] >= seq;
        gm = umax(umax(a0[1], a0[3]), umax(a1[1], a1[3]));
      }
      if (__builtin_amdgcn_ballot_w64(!done) == 0) break;
      if (expired(t0, tmo_ptr, seq)) { gm = 0x7fffffffu; break; }
    }
    u32x2 gr = {seq, wave_umax(gm)};
    u32 first = 0;
    if (ln == 0) first = __hip_atomic_fetch_add(&red[3], 1u, __ATOMIC_RELAXED, __HIP_MEMORY_SCOPE_WORKGROUP);
    if (rfl(first) == 0) {
#pragma unroll
      for (int k = 0; k < NC / 8; ++k)
        asm volatile("global_store_dwordx2 %0, %1, off sc0 sc1" :: "v"(peer + AGR_OFF + (k * 8 + (ln >> 3)) * 256 + rank * 8), "v"(gr) : "memory");
    }
  }
  if (closer) {
    const char* gp = myws + AGR_OFF + (b % NC) * 256 + (ln & 7) * 8;
    u32x2 f;
    for (;;) {
      asm volatile("global_load_dwordx2 %0, %1, off sc0 sc1\n\ts_waitcnt vmcnt(0)" : "=&v"(f) : "v"(gp) : "memory");
      if (__builtin_amdgcn_ballot_w64(f[0] < seq) == 0) break;
      if (expired(t0, tmo_ptr, seq)) { f[1] = 0x7fffffffu; break; }
      if (CFG_SLP2) __builtin_amdgcn_s_sleep(CFG_SLP2);
    }
    u32 gm = f[1];
    gm = umax(gm, dpp0<0x111, 0xf>(gm)); gm = umax(gm, dpp0<0x112, 0xf>(gm)); gm = umax(gm, dpp0<0x114, 0xf>(gm));
    if (ln == 7) red[8] = gm;
  }
  STAMP(8);
  wg_barrier();
  STAMP(9);
  const float gmax = __builtin_bit_cast(float, red[8]);
  const float inv = 448.0f / gmax * unsc;
  auto q4 = [&](const f32x16& a, int d) -> u32 {
    int t = __builtin_amdgcn_cvt_pk_fp8_f32(a[4 * d] * inv, a[4 * d + 1] * inv, 0, false);
    return (u32)__builtin_amdgcn_cvt_pk_fp8_f32(a[4 * d + 2] * inv, a[4 * d + 3] * inv, t, true);
  };
  {
    __amdgpu_buffer_rsrc_t ro = __builtin_amdgcn_make_buffer_rsrc(outp, 0, NR * MLOC * NLOC, 0x00020000);
    const u32 vo = (g * MLOC + (mbi ^ 1) * 32 + (ln & 15)) * NLOC + c * 128 + (ln >> 4) * 16;
    auto put = [&](const f32x16& a, u32 off) {
      const u32x2 e = __builtin_amdgcn_permlane32_swap(q4(a, 0), q4(a, 2), false, false);
      const u32x2 f = __builtin_amdgcn_permlane32_swap(q4(a, 1), q4(a, 3), false, false);
      const u32x2 t01 = __builtin_amdgcn_permlane16_swap(e[0], f[0], false, false);
      const u32x2 t23 = __builtin_amdgcn_permlane16_swap(e[1], f[1], false, false);
      const u32x4 o = {t01[0], t01[1], t23[0], t23[1]};
      __builtin_amdgcn_raw_buffer_store_b128(o, ro, vo + off, 0, 2);
    };
    put(acc0, 0); put(acc1, 64); put(acc2, 16 * NLOC); put(acc3, 16 * NLOC + 64);
  }
  if (v == 0 && ln == 0) __hip_atomic_store(cnt_ptr, seq, __ATOMIC_RELAXED, __HIP_MEMORY_SCOPE_SYSTEM);
}

}

extern "C" void kernel_launch(int rank, int nranks, ncclComm_t comm, hipStream_t stream,
                              void* const* d_in, const int* in_sizes, int n_in,
                              void* d_out, int out_size, const int* in_dtypes, int out_dtype,
                              void* const* d_out_all, void* const* d_ws_all, size_t ws_bytes) {
  (void)comm; (void)in_dtypes; (void)out_dtype; (void)d_out_all;
  if (nranks != NR || (unsigned)rank >= (unsigned)NR || n_in < 2 || ws_bytes < WS_NEED ||
      in_sizes[0] != MLOC * KD || in_sizes[1] != KD * NLOC || out_size != NR * MLOC * NLOC) {
    fprintf(stderr, "unexpected problem shape\n");
    return;
  }
  Params p;
  {
    int nearx = 0, farx = 4;
    p.order = 0;
    for (int r = 0; r < NR; ++r) {
      int xcd = (int)SELF_XCD;
      if (r != rank) {
        if (((GROUP_BITS >> r) & 1) != ((GROUP_BITS >> rank) & 1)) xcd = farx++;
        else { if (nearx == (int)SELF_XCD) ++nearx; xcd = nearx++; }
      }
      p.ws[xcd] = (char*)d_ws_all[r];
      p.order |= (u32)r << (4 * xcd);
    }
  }
  p.myws = (char*)d_ws_all[rank];
  p.x = (const char*)d_in[0];
  p.w = (const char*)d_in[1];
  p.out = (char*)d_out;
  p.rank = (u32)rank;
  hipLaunchKernelGGL(ag_gemm_q, dim3(NWG), dim3(512), 0, stream, p);
  HIP_CHECK(hipGetLastError());
}
```
